```python
import numpy as np

kernel_name = "gpu_gemm_a2a_m4096_k4096_n4096_bf16_q8"


ABSMAX_RTOL = 0.02

COMPUTE_DTYPE = "bfloat16"
EPILOGUE = "quant_int8"
IN_QUANT = None
OUT_QUANT = "int8"
IN_DTYPES = {"x": "bfloat16", "w_mat": "bfloat16"}
WORLD_SIZE = 8

def _to_bf16(x):
    u = np.ascontiguousarray(x, dtype=np.float32).view(np.uint32)
    return ((u + 0x7FFF + ((u >> 16) & 1)) & 0xFFFF0000).view(np.float32)

def setup_inputs():

    x = _to_bf16(np.random.randn(4096, 4096).astype(np.float32))
    w_mat = _to_bf16((np.random.randn(4096, 4096) / np.sqrt(4096)).astype(np.float32))
    return {"x": x, "w_mat": w_mat}

def _quant_dequant(y, amax):
    scale = amax / 127.0
    q = np.clip(np.round(y / scale), -127.0, 127.0)
    return (q * scale).astype(np.float32)
def _epilogue(y):
    return _quant_dequant(y, np.abs(y).max())

def reference(x, w_mat):
    y = (x @ w_mat).astype(np.float32)
    return _epilogue(y)
```

```cpp
#include <hip/hip_runtime.h>
#include <rccl/rccl.h>
#include <cstdio>

#define HIP_CHECK(expr) do { hipError_t _e = (expr); if (_e != hipSuccess) { \
    fprintf(stderr, "HIP error %s at %s:%d\n", hipGetErrorString(_e), __FILE__, __LINE__); \
    return; } } while (0)

namespace {

typedef short s16x4 __attribute__((ext_vector_type(4)));
typedef short s16x8 __attribute__((ext_vector_type(8)));
typedef float f32x4 __attribute__((ext_vector_type(4)));
typedef unsigned u32x4 __attribute__((ext_vector_type(4)));
#define LAS __attribute__((address_space(3)))
#define GAS __attribute__((address_space(1)))

constexpr int NR = 8;
constexpr int MLOC = 512, N = 4096, K = 4096, NLOC = 512;
constexpr int BM = 128, BN = 128, BK = 64, KS = 2;
constexpr int T = K / KS / BK;
constexpr int S = 4;
constexpr int A_BYTES = BM * BK * 2, B_BYTES = BK * BN * 2, SLOT = A_BYTES + B_BYTES;
constexpr int LDS_MISC = S * SLOT;
constexpr int LDS_BYTES = S * SLOT + 256;
constexpr int NWG = 256;
constexpr int PER = 8;

constexpr size_t WS_TMO = 0;
constexpr size_t WS_CNT = 4096;
constexpr size_t WS_AMAX = 65536;
constexpr size_t WS_DBG = 262144;
constexpr size_t WS_XCH = 1u << 20;
constexpr size_t WS_NEED = WS_XCH + (size_t)NWG * 32768;
constexpr unsigned long long DEADLINE = 50000000ull;

struct Params {
  const void* x; const void* w;
  char* out[NR]; char* ws[NR];
  int rank;
};

#define MEMFENCE asm volatile("" ::: "memory")
#define SB0 __builtin_amdgcn_sched_barrier(0)
#define BAR do { MEMFENCE; __builtin_amdgcn_s_barrier(); MEMFENCE; } while (0)
#define MF(acc, a, b) asm volatile("v_mfma_f32_16x16x32_bf16 %0, %1, %2, %0" : "+v"(acc) : "v"(a), "v"(b))
template <int N_> __device__ __forceinline__ void wait_vm() { asm volatile("s_waitcnt vmcnt(%0)" :: "i"(N_) : "memory"); }
__device__ __forceinline__ void wait_lgkm0() { MEMFENCE; __builtin_amdgcn_s_waitcnt(0xC07F); MEMFENCE; }
__device__ __forceinline__ int hB(int row) { return ((row & 3) | (((row >> 3) & 1) << 2)) << 1; }
constexpr int AUX_AGENT = 16, AUX_SYS = 17;

#define STAMP(i) do { } while (0)

__device__ __forceinline__ float wave_max(float f) {
  unsigned v = __float_as_uint(f);
  v = max(v, (unsigned)__builtin_amdgcn_update_dpp(0, (int)v, 0xB1, 0xf, 0xf, false));
  v = max(v, (unsigned)__builtin_amdgcn_update_dpp(0, (int)v, 0x4E, 0xf, 0xf, false));
  v = max(v, (unsigned)__builtin_amdgcn_update_dpp(0, (int)v, 0x141, 0xf, 0xf, false));
  v = max(v, (unsigned)__builtin_amdgcn_update_dpp(0, (int)v, 0x140, 0xf, 0xf, false));
  const unsigned a = __builtin_amdgcn_readlane(v, 0), b = __builtin_amdgcn_readlane(v, 16);
  const unsigned c = __builtin_amdgcn_readlane(v, 32), d = __builtin_amdgcn_readlane(v, 48);
  return __uint_as_float(max(max(a, b), max(c, d)));
}

__device__ __forceinline__ bool give_up(GAS unsigned* tmo, unsigned call, unsigned long long t0, unsigned& spins) {
  __builtin_amdgcn_s_sleep(1);
  if ((++spins & 15u) != 0u) return false;
  if ((unsigned)__builtin_amdgcn_readfirstlane(__hip_atomic_load(tmo, __ATOMIC_RELAXED, __HIP_MEMORY_SCOPE_AGENT)) == call) return true;
  if (__builtin_amdgcn_s_memrealtime() - t0 > DEADLINE) {
    __hip_atomic_store(tmo, call, __ATOMIC_RELAXED, __HIP_MEMORY_SCOPE_AGENT);
    return true;
  }
  return false;
}

__global__ __launch_bounds__(512, 2) void gemm_a2a_q8(const Params p) {
#if defined(__HIP_DEVICE_COMPILE__)
  extern __shared__ __attribute__((aligned(16))) char smem[];
  LAS char* lds = (LAS char*)smem;
  LAS float* s_wmax = (LAS float*)(lds + LDS_MISC);
  LAS unsigned* s_call = (LAS unsigned*)(lds + LDS_MISC + 64);
  const int bid = blockIdx.x;
  const int xcd = bid & 7, idx = bid >> 3;
  const int ks = xcd & 1, tm = idx & 3, tn = idx >> 2;
  const int m0 = tm * BM, n0 = (xcd >> 1) * 1024 + tn * BN;
  const int kbase = ks * (K / KS);
  const int wid = __builtin_amdgcn_readfirstlane(threadIdx.x >> 6), lane = threadIdx.x & 63;
  const int rank = p.rank;
  GAS char* myws = (GAS char*)p.ws[rank];
  GAS unsigned* tmo = (GAS unsigned*)(myws + WS_TMO);

  if (wid >= 4) {
    const int lw = wid - 4;
    auto rx = __builtin_amdgcn_make_buffer_rsrc((void*)p.x, 0, MLOC * K * 2, 0x00020000);
    auto rw = __builtin_amdgcn_make_buffer_rsrc((void*)p.w, 0, K * N * 2, 0x00020000);
    int voA[4], voB[4];
#pragma unroll
    for (int i = 0; i < 4; ++i) {
      const int row = 8 * (lw * 4 + i) + (lane >> 3), ch = (lane & 7) ^ ((row >> 1) & 7);
      voA[i] = (m0 + (row ^ (ks * 32))) * (K * 2) + kbase * 2 + ch * 16;
    }
#pragma unroll
    for (int i = 0; i < 4; ++i) {
      const int krow = 4 * (lw * 4 + i) + (lane >> 4), ch = (lane & 15) ^ hB(krow);
      voB[i] = (kbase + krow) * (N * 2) + n0 * 2 + ch * 16;
    }
    auto issue = [&](int kt, int slot) __attribute__((always_inline)) {
      const int soA = kt * (BK * 2), soB = kt * (BK * N * 2);
      LAS char* sb = lds + slot * SLOT;
#pragma unroll
      for (int i = 0; i < 4; ++i) {
        __builtin_amdgcn_raw_ptr_buffer_load_lds(rx, (LAS void*)(sb + (lw * 4 + i) * 1024), 16, voA[i], soA, 0, 0);
        __builtin_amdgcn_raw_ptr_buffer_load_lds(rw, (LAS void*)(sb + A_BYTES + (lw * 4 + i) * 1024), 16, voB[i], soB, 0, 0);
      }
    };
    STAMP(0);
    issue(0, 0); issue(1, 1);
    wait_vm<PER>(); BAR;
    issue(2, 2); issue(3, 3);
    wait_vm<2 * PER>(); BAR;
    int slot = 0;
    for (int t = 1; t <= T - S; ++t) {
      issue(t + S - 1, slot);
      slot = (slot + 1) & (S - 1);
      wait_vm<2 * PER>(); BAR;
    }
    wait_vm<PER>(); BAR;
    wait_vm<0>(); BAR;
    GAS unsigned long long* gdst = (GAS unsigned long long*)((GAS char*)p.ws[lane & 7] + WS_AMAX);
    BAR;
    const unsigned call = __builtin_amdgcn_readfirstlane(s_call[0]);
    const unsigned par = call & 1u;
    if (lw == 0) {
      const float m = fmaxf(fmaxf(s_wmax[0], s_wmax[1]), fmaxf(s_wmax[2], s_wmax[3]));
      const unsigned long long g = ((unsigned long long)call << 32) | __float_as_uint(m);
      if (lane < NR) __hip_atomic_store(gdst + par * 2048 + rank * NWG + bid, g, __ATOMIC_RELAXED, __HIP_MEMORY_SCOPE_SYSTEM);
      STAMP(4);
    }
    auto ra = __builtin_amdgcn_make_buffer_rsrc((void*)(p.ws[rank] + WS_AMAX), 0, 2 * 2048 * 8, 0x00020000);
    float gm = 0.f; bool bad = false;
    {
      const unsigned long long t0 = __builtin_amdgcn_s_memrealtime(); unsigned spins = 0;
      const int vo = par * 16384 + lw * 4096 + lane * 16;
      for (;;) {
        u32x4 v[4];
#pragma unroll
        for (int i = 0; i < 4; ++i) v[i] = __builtin_amdgcn_raw_buffer_load_b128(ra, vo + i * 1024, 0, AUX_SYS);
        bool ok = true; gm = 0.f;
#pragma unroll
        for (int i = 0; i < 4; ++i) {
          ok = ok && v[i][1] == call && v[i][3] == call;
          gm = fmaxf(gm, fmaxf(__uint_as_float(v[i][0]), __uint_as_float(v[i][2])));
        }
        if (__all(ok)) break;
        if (give_up(tmo, call, t0, spins)) { bad = true; break; }
      }
    }
    gm = wave_max(gm);
    if (lane == 0) { s_wmax[4 + lw] = gm; s_call[5 + lw] = bad ? 1u : 0u; }
    wait_lgkm0();
    if (lw == 0) STAMP(5);
    BAR;
    auto ro = __builtin_amdgcn_make_buffer_rsrc((void*)p.out[rank], 0, N * NLOC, 0x00020000);
    {
      const unsigned long long t0 = __builtin_amdgcn_s_memrealtime(); unsigned spins = 0;
      const int vo = bid * 8192 + lw * 2048 + lane * 16;
      for (;;) {
        u32x4 v[2];
#pragma unroll
        for (int i = 0; i < 2; ++i) v[i] = __builtin_amdgcn_raw_buffer_load_b128(ro, vo + i * 1024, 0, AUX_SYS);
        unsigned z = 0;
#pragma unroll
        for (int i = 0; i < 2; ++i)
#pragma unroll
          for (int j = 0; j < 4; ++j) { const unsigned t = v[i][j] ^ 0x80808080u; z |= (t - 0x01010101u) & ~t & 0x80808080u; }
        if (__all(z == 0u)) break;
        if (give_up(tmo, call, t0, spins)) break;
      }
    }
    if (lw == 0) STAMP(7);
  } else {
    const int wr = wid >> 1, wc = wid & 1;
    const int g = lane >> 4, r = lane & 15, q = r >> 2, pp = lane & 3;
    {
      auto ro = __builtin_amdgcn_make_buffer_rsrc((void*)p.out[rank], 0, N * NLOC, 0x00020000);
      const u32x4 f = {0x80808080u, 0x80808080u, 0x80808080u, 0x80808080u};
      const int vo = bid * 8192 + wid * 2048 + lane * 16;
      __builtin_amdgcn_raw_buffer_store_b128(f, ro, vo, 0, AUX_SYS);
      __builtin_amdgcn_raw_buffer_store_b128(f, ro, vo + 1024, 0, AUX_SYS);
    }
    GAS unsigned* cnt = (GAS unsigned*)(myws + WS_CNT + bid * 64);
    const unsigned callv = __hip_atomic_load(cnt, __ATOMIC_RELAXED, __HIP_MEMORY_SCOPE_AGENT);
    unsigned base[6], cur[6];
#pragma unroll
    for (int k2 = 0; k2 < 2; ++k2) base[k2] = (wr * 64 + r) * 128 + (((4 * k2 + g) ^ ((r >> 1) & 7)) * 16);
    {
      const int krow = 8 * g + q, hx = hB(krow);
#pragma unroll
      for (int nb = 0; nb < 4; ++nb) base[2 + nb] = A_BYTES + krow * 256 + (((wc * 8 + 2 * nb + (pp >> 1)) ^ hx) * 16) + (pp & 1) * 8;
    }
    f32x4 acc[4][4];
#pragma unroll
    for (int i = 0; i < 4; ++i)
#pragma unroll
      for (int j = 0; j < 4; ++j) acc[i][j] = (f32x4){0.f, 0.f, 0.f, 0.f};
    s16x8 fa[2][2][4];
    s16x4 fb[2][2][4][2];
    auto rd1 = [&](int buf, int i) __attribute__((always_inline)) {
      const int k2 = i / 12, j = i % 12;
      if (j < 8) fb[buf][k2][j >> 1][j & 1] = __builtin_amdgcn_ds_read_tr16_b64_v4i16((LAS s16x4*)(lds + cur[2 + (j >> 1)] + (32 * k2 + 4 * (j & 1)) * 256));
      else fa[buf][k2][j - 8] = *(LAS s16x8*)(lds + cur[k2] + (j - 8) * 2048);
    };
    auto mf1 = [&](int buf, int i) __attribute__((always_inline)) {
      const int k2 = i >> 4, mb = (i >> 2) & 3, nb = i & 3;
      s16x8 b8 = __builtin_shufflevector(fb[buf][k2][nb][0], fb[buf][k2][nb][1], 0, 1, 2, 3, 4, 5, 6, 7);
      MF(acc[mb][nb], b8, fa[buf][k2][mb]);
    };
    int slot = 0;
    auto advance = [&]() __attribute__((always_inline)) {
      slot = (slot + 1) & (S - 1);
      const unsigned so = slot * SLOT;
#pragma unroll
      for (int i = 0; i < 6; ++i) cur[i] = base[i] + so;
    };
    auto stage = [&](int buf) __attribute__((always_inline)) {
      BAR; SB0;
#pragma unroll
      for (int i = 0; i < 32; ++i) {
        mf1(buf ^ 1, i);
        if (i < 24) rd1(buf, i);
        if (i == 28) advance();
        if (i == 30) wait_lgkm0();
        SB0;
      }
    };
#pragma unroll
    for (int i = 0; i < 6; ++i) cur[i] = base[i];
    BAR; SB0;
    if (wid == 0) STAMP(1);
#pragma unroll
    for (int i = 0; i < 24; ++i) rd1(0, i);
    advance();
    wait_lgkm0(); SB0;
    for (int t = 1; t < T - 1; t += 2) { stage(1); stage(0); }
    stage(1);
    auto rxch = __builtin_amdgcn_make_buffer_rsrc((void*)(p.ws[rank] + WS_XCH), 0, NWG * 32768, 0x00020000);
    const int vos = (bid ^ 1) * 32768 + wid * 8192 + lane * 16;
    SB0;
#pragma unroll
    for (int i = 0; i < 32; ++i) if (((i >> 2) & 3) >= 2) mf1(1, i);
    SB0;
#pragma unroll
    for (int u = 0; u < 16; ++u) {
      mf1(1, (u >> 3) * 16 + (u & 7));
      SB0;
      if (u >= 6 && u < 14) {
        const int i = (u - 6) >> 2, nb = (u - 6) & 3;
        u32x4 e;
#pragma unroll
        for (int j = 0; j < 4; ++j) e[j] = max(__float_as_uint(acc[2 + i][nb][j]) ^ 0x80000000u, 1u);
        __builtin_amdgcn_raw_buffer_store_b128(e, rxch, vos + (i * 4 + nb) * 1024, 0, AUX_AGENT);
        SB0;
      }
    }
    asm volatile("s_nop 15\n\ts_nop 7" ::: "memory");
    SB0;
    if (wid == 0) STAMP(2);
    const unsigned call = __builtin_amdgcn_readfirstlane(callv) + 1u;
    bool bad = false;
    const int vox = bid * 32768 + wid * 8192 + lane * 16;
    {
      const unsigned long long t0 = __builtin_amdgcn_s_memrealtime(); unsigned spins = 0;
      for (;;) {
        u32x4 v[8];
#pragma unroll
        for (int i = 0; i < 8; ++i) v[i] = __builtin_amdgcn_raw_buffer_load_b128(rxch, vox + i * 1024, 0, AUX_AGENT);
        unsigned mn = 0xffffffffu;
#pragma unroll
        for (int i = 0; i < 8; ++i) mn = min(min(min(v[i][0], v[i][1]), min(v[i][2], v[i][3])), mn);
        if (__all(mn != 0u)) {
#pragma unroll
          for (int i = 0; i < 8; ++i)
#pragma unroll
            for (int j = 0; j < 4; ++j) acc[i >> 2][i & 3][j] -= __uint_as_float(v[i][j]);
          break;
        }
        if (give_up(tmo, call, t0, spins)) { bad = true; break; }
      }
    }
    float m = 0.f;
#pragma unroll
    for (int i = 0; i < 2; ++i)
#pragma unroll
      for (int nb = 0; nb < 4; ++nb)
#pragma unroll
        for (int j = 0; j < 4; ++j) m = fmaxf(m, fabsf(acc[i][nb][j]));
    m = wave_max(m);
    if (lane == 0) { s_wmax[wid] = m; s_call[1 + wid] = bad ? 1u : 0u; if (wid == 0) s_call[0] = call; }
    wait_lgkm0();
    if (wid == 0) STAMP(3);
    BAR;
    {
      const u32x4 z = {0u, 0u, 0u, 0u};
#pragma unroll
      for (int i = 0; i < 8; ++i) __builtin_amdgcn_raw_buffer_store_b128(z, rxch, vox + i * 1024, 0, AUX_AGENT);
      if (threadIdx.x == 0) __hip_atomic_store(cnt, call, __ATOMIC_RELAXED, __HIP_MEMORY_SCOPE_AGENT);
    }
    const unsigned dest = (unsigned)n0 / NLOC;
    auto rd = __builtin_amdgcn_make_buffer_rsrc((void*)p.out[dest & (NR - 1)], 0, N * NLOC, 0x00020000);
    const int vod = (rank * MLOC + m0 + wr * 64 + ks * 32 + r) * NLOC + (n0 & (NLOC - 1)) + wc * 64 + g * 16;
    BAR;
    const float amax = fmaxf(fmaxf(s_wmax[4], s_wmax[5]), fmaxf(s_wmax[6], s_wmax[7]));
    unsigned anybad = 0;
#pragma unroll
    for (int i = 1; i <= 8; ++i) anybad |= s_call[i];
    const float scale = amax / 127.0f;
    const float inv = scale > 0.f ? 1.0f / scale : 0.f;
    constexpr float MAGIC = 12582912.0f;
#pragma unroll
    for (int i = 0; i < 2; ++i) {
      unsigned d[4];
#pragma unroll
      for (int nb = 0; nb < 4; ++nb) {
        unsigned b[4];
#pragma unroll
        for (int j = 0; j < 4; ++j)
          b[j] = __float_as_uint(__builtin_amdgcn_fmed3f(fmaf(acc[i][nb][j], inv, MAGIC), MAGIC - 127.0f, MAGIC + 127.0f));
        d[nb] = __builtin_amdgcn_perm(__builtin_amdgcn_perm(b[3], b[2], 0x0c0c0400u), __builtin_amdgcn_perm(b[1], b[0], 0x0c0c0400u), 0x05040100u);
      }
      { auto t = __builtin_amdgcn_permlane32_swap(d[0], d[2], false, false); d[0] = t[0]; d[2] = t[1]; }
      { auto t = __builtin_amdgcn_permlane32_swap(d[1], d[3], false, false); d[1] = t[0]; d[3] = t[1]; }
      { auto t = __builtin_amdgcn_permlane16_swap(d[0], d[1], false, false); d[0] = t[0]; d[1] = t[1]; }
      { auto t = __builtin_amdgcn_permlane16_swap(d[2], d[3], false, false); d[2] = t[0]; d[3] = t[1]; }
      u32x4 o = {d[0], d[1], d[2], d[3]};
      if (anybad) o = (u32x4){0x80808080u, 0x80808080u, 0x80808080u, 0x80808080u};
      __builtin_amdgcn_raw_buffer_store_b128(o, rd, vod + i * 16 * NLOC, 0, AUX_SYS);
    }
    if (wid == 0) STAMP(6);
  }
#endif
}

}

extern "C" void kernel_launch(int rank, int nranks, ncclComm_t comm,
                              hipStream_t stream,
                              void* const* d_in, const int* in_sizes, int n_in,
                              void* d_out, int out_size,
                              const int* in_dtypes, int out_dtype,
                              void* const* d_out_all,
                              void* const* d_ws_all, size_t ws_bytes) {
  (void)comm; (void)in_dtypes; (void)out_dtype; (void)d_out;
  if (nranks != NR || n_in != 2 || in_sizes[0] != MLOC * K || in_sizes[1] != K * N || out_size != N * NLOC ||
      ws_bytes < WS_NEED || (unsigned)rank >= (unsigned)NR) {
    fprintf(stderr, "kernel_launch: unexpected problem shape\n");
    return;
  }
  static int ready = 0;
  if (!ready) {
    HIP_CHECK(hipFuncSetAttribute((const void*)gemm_a2a_q8, hipFuncAttributeMaxDynamicSharedMemorySize, LDS_BYTES));
    ready = 1;
  }
  Params p;
  p.x = d_in[0]; p.w = d_in[1]; p.rank = rank;
  for (int i = 0; i < NR; ++i) { p.out[i] = (char*)d_out_all[i]; p.ws[i] = (char*)d_ws_all[i]; }
  hipLaunchKernelGGL(gemm_a2a_q8, dim3(NWG), dim3(512), LDS_BYTES, stream, p);
  HIP_CHECK(hipGetLastError());
}
```

```cpp
#include <hip/hip_runtime.h>
#include <rccl/rccl.h>
#include <cstdio>

#define HIP_CHECK(expr) do { hipError_t _e = (expr); if (_e != hipSuccess) { \
    fprintf(stderr, "HIP error %s at %s:%d\n", hipGetErrorString(_e), __FILE__, __LINE__); \
    return; } } while (0)

namespace {

typedef short s16x2 __attribute__((ext_vector_type(2)));
typedef short s16x4 __attribute__((ext_vector_type(4)));
typedef short s16x8 __attribute__((ext_vector_type(8)));
typedef float f32x4 __attribute__((ext_vector_type(4)));
typedef unsigned u32x2 __attribute__((ext_vector_type(2)));
typedef unsigned u32x4 __attribute__((ext_vector_type(4)));
#define LAS __attribute__((address_space(3)))
#define GAS __attribute__((address_space(1)))

constexpr int NR = 8;
constexpr int MLOC = 512, N = 4096, K = 4096, NLOC = 512;
constexpr int BM = 128, BN = 128, BK = 64, KS = 2;
constexpr int T = K / KS / BK;
constexpr int S = 4;
constexpr int A_BYTES = BM * BK * 2, B_BYTES = BK * BN * 2, SLOT = A_BYTES + B_BYTES;
constexpr int LDS_MISC = S * SLOT;
constexpr int LDS_BYTES = S * SLOT + 256;
constexpr int NWG = 256;
constexpr int PER = 8;

constexpr size_t WS_TMO = 0;
constexpr size_t WS_CNT = 4096;
constexpr size_t WS_AMAX = 65536;
constexpr size_t WS_DBG = 262144;
constexpr size_t WS_XCH = 1u << 20;
constexpr int XCH_WG = 32768;
constexpr size_t WS_STG = WS_XCH + (size_t)NWG * XCH_WG;
constexpr int STG_PAR = 2u << 20;
constexpr size_t WS_NEED = WS_STG + 2 * (size_t)STG_PAR;
constexpr unsigned long long DEADLINE = 50000000ull;

#define PF_MODE 5
#define PF_LEAD 3
#define POLL_SLEEP 4
#define XCH16 1
#define EARLY 1

struct Params {
  const void* x; const void* w;
  char* myout; char* myws;
  int rank, pad;
  char* out[NR]; char* ws[NR];
};

#define MEMFENCE asm volatile("" ::: "memory")
#define SB0 __builtin_amdgcn_sched_barrier(0)
#define BAR do { MEMFENCE; __builtin_amdgcn_s_barrier(); MEMFENCE; } while (0)
#define MF(acc, a, b) asm volatile("v_mfma_f32_16x16x32_bf16 %0, %1, %2, %0" : "+v"(acc) : "v"(a), "v"(b))
template <int N_> __device__ __forceinline__ void wait_vm() { asm volatile("s_waitcnt vmcnt(%0)" :: "i"(N_) : "memory"); }
__device__ __forceinline__ void wait_lgkm0() { MEMFENCE; __builtin_amdgcn_s_waitcnt(0xC07F); MEMFENCE; }
__device__ __forceinline__ int hB(int row) { return ((row & 3) | (((row >> 3) & 1) << 2)) << 1; }
constexpr int AUX_AGENT = 16, AUX_SYS = 17;
constexpr float MAGIC = 12582912.0f;

#define STAMP(i) do { } while (0)

__device__ __forceinline__ float wave_max(float f) {
  unsigned v = __float_as_uint(f);
  v = max(v, (unsigned)__builtin_amdgcn_update_dpp(0, (int)v, 0xB1, 0xf, 0xf, false));
  v = max(v, (unsigned)__builtin_amdgcn_update_dpp(0, (int)v, 0x4E, 0xf, 0xf, false));
  v = max(v, (unsigned)__builtin_amdgcn_update_dpp(0, (int)v, 0x141, 0xf, 0xf, false));
  v = max(v, (unsigned)__builtin_amdgcn_update_dpp(0, (int)v, 0x140, 0xf, 0xf, false));
  const unsigned a = __builtin_amdgcn_readlane(v, 0), b = __builtin_amdgcn_readlane(v, 16);
  const unsigned c = __builtin_amdgcn_readlane(v, 32), d = __builtin_amdgcn_readlane(v, 48);
  return __uint_as_float(max(max(a, b), max(c, d)));
}

__device__ __forceinline__ bool give_up(GAS unsigned* tmo, unsigned call, unsigned long long t0, unsigned& spins) {
  __builtin_amdgcn_s_sleep(POLL_SLEEP);
  if ((++spins & 15u) != 0u) return false;
  if ((unsigned)__builtin_amdgcn_readfirstlane(__hip_atomic_load(tmo, __ATOMIC_RELAXED, __HIP_MEMORY_SCOPE_AGENT)) == call) return true;
  if (__builtin_amdgcn_s_memrealtime() - t0 > DEADLINE) {
    __hip_atomic_store(tmo, call, __ATOMIC_RELAXED, __HIP_MEMORY_SCOPE_AGENT);
    return true;
  }
  return false;
}
__device__ __forceinline__ unsigned enc16(float a, float b, unsigned tag) {
  return (__builtin_bit_cast(unsigned, __builtin_amdgcn_cvt_pknorm_i16(a, b)) & ~1u) | tag;
}
__device__ __forceinline__ float lo16(unsigned v) { return (float)(short)(v & 0xffffu); }
__device__ __forceinline__ float hi16(unsigned v) { return (float)((int)v >> 16); }
__device__ __forceinline__ unsigned tagbits(const u32x4& v, unsigned pm) { return (v[0] ^ pm) | (v[1] ^ pm) | (v[2] ^ pm) | (v[3] ^ pm); }
__device__ __forceinline__ unsigned q4(float a, float b, float c, float d, float f) {
  const unsigned b0 = __float_as_uint(__builtin_amdgcn_fmed3f(fmaf(a, f, MAGIC), MAGIC - 127.0f, MAGIC + 127.0f));
  const unsigned b1 = __float_as_uint(__builtin_amdgcn_fmed3f(fmaf(b, f, MAGIC), MAGIC - 127.0f, MAGIC + 127.0f));
  const unsigned b2 = __float_as_uint(__builtin_amdgcn_fmed3f(fmaf(c, f, MAGIC), MAGIC - 127.0f, MAGIC + 127.0f));
  const unsigned b3 = __float_as_uint(__builtin_amdgcn_fmed3f(fmaf(d, f, MAGIC), MAGIC - 127.0f, MAGIC + 127.0f));
  return __builtin_amdgcn_perm(__builtin_amdgcn_perm(b3, b2, 0x0c0c0400u), __builtin_amdgcn_perm(b1, b0, 0x0c0c0400u), 0x05040100u);
}
__device__ __forceinline__ u32x4 xpose(unsigned d0, unsigned d1, unsigned d2, unsigned d3) {
  { auto t = __builtin_amdgcn_permlane32_swap(d0, d2, false, false); d0 = t[0]; d2 = t[1]; }
  { auto t = __builtin_amdgcn_permlane32_swap(d1, d3, false, false); d1 = t[0]; d3 = t[1]; }
  { auto t = __builtin_amdgcn_permlane16_swap(d0, d1, false, false); d0 = t[0]; d1 = t[1]; }
  { auto t = __builtin_amdgcn_permlane16_swap(d2, d3, false, false); d2 = t[0]; d3 = t[1]; }
  return (u32x4){d0, d1, d2, d3};
}

__global__ __launch_bounds__(512, 2) void gemm_a2a_q8(const Params p) {
#if defined(__HIP_DEVICE_COMPILE__)
  extern __shared__ __attribute__((aligned(16))) char smem[];
  LAS char* lds = (LAS char*)smem;
  LAS float* s_wmax = (LAS float*)(lds + LDS_MISC);
  LAS unsigned* s_call = (LAS unsigned*)(lds + LDS_MISC + 16);
  volatile LAS u32x4* s_g = (volatile LAS u32x4*)(lds + LDS_MISC + 32);
  const int bid = blockIdx.x;
  const int xcd = bid & 7, idx = bid >> 3;
  const int ks = xcd & 1, tm = idx & 3, tn = idx >> 2;
  const int m0 = tm * BM, n0 = (xcd >> 1) * 1024 + tn * BN;
  const int kbase = ks * (K / KS);
  const int wid = __builtin_amdgcn_readfirstlane(threadIdx.x >> 6), lane = threadIdx.x & 63;
  const int rank = p.rank;
  GAS char* myws = (GAS char*)p.myws;
  GAS unsigned* tmo = (GAS unsigned*)(myws + WS_TMO);
  auto wait_amax = [&](bool& bad) __attribute__((always_inline)) -> float {
    u32x4 g;
    for (;;) { g = *s_g; if (g[1] != 0u && g[3] != 0u) break; __builtin_amdgcn_s_sleep(1); }
    bad = bad || g[1] != 1u || g[3] != 1u;
    const float scale = fmaxf(__uint_as_float(g[0]), __uint_as_float(g[2])) / 127.0f;
    return scale > 0.f ? 1.0f / scale : 0.f;
  };

  if (wid >= 4) {
    const int lw = wid - 4;
    auto rx = __builtin_amdgcn_make_buffer_rsrc((void*)p.x, 0, MLOC * K * 2, 0x00020000);
    auto rw = __builtin_amdgcn_make_buffer_rsrc((void*)p.w, 0, K * N * 2, 0x00020000);
    int voA[4], voB[4];
#pragma unroll
    for (int i = 0; i < 4; ++i) {
      const int row = 8 * (lw * 4 + i) + (lane >> 3), ch = (lane & 7) ^ ((row >> 1) & 7);
      voA[i] = (m0 + (row ^ (ks * 32))) * (K * 2) + kbase * 2 + ch * 16;
    }
#pragma unroll
    for (int i = 0; i < 4; ++i) {
      const int krow = 4 * (lw * 4 + i) + (lane >> 4), ch = (lane & 15) ^ hB(krow);
      voB[i] = (kbase + krow) * (N * 2) + n0 * 2 + ch * 16;
    }
    auto issue = [&](int kt, int slot) __attribute__((always_inline)) {
      const int soA = kt * (BK * 2), soB = kt * (BK * N * 2);
      LAS char* sb = lds + slot * SLOT;
#pragma unroll
      for (int i = 0; i < 4; ++i) {
        __builtin_amdgcn_raw_ptr_buffer_load_lds(rx, (LAS void*)(sb + (lw * 4 + i) * 1024), 16, voA[i], soA, 0, 0);
        __builtin_amdgcn_raw_ptr_buffer_load_lds(rw, (LAS void*)(sb + A_BYTES + (lw * 4 + i) * 1024), 16, voB[i], soB, 0, 0);
      }
    };
    unsigned pfd = 0;
    const int pg = idx >> 2, pj = (idx & 3) * 4 + lw;
    const int qvw = (kbase + pj * 4 + (lane >> 4)) * (N * 2) + (xcd >> 1) * 2048 + (lane & 15) * 128;
    const int qvx = (pj * 32 + (lane & 31)) * (K * 2) + kbase * 2;
    auto burst = [&](int kt) __attribute__((always_inline)) {
      asm volatile("buffer_load_dword %0, %1, %2, %3 offen" : "+v"(pfd) : "v"(qvw), "s"(rw), "s"(kt * (BK * N * 2)));
      asm volatile("buffer_load_dword %0, %1, %2, %3 offen" : "+v"(pfd) : "v"(qvx), "s"(rx), "s"(kt * (BK * 2)));
    };
    STAMP(0);
    issue(0, 0); issue(1, 1);
    wait_vm<PER>(); BAR;
    issue(2, 2); issue(3, 3);
    unsigned long long gd = (unsigned long long)p.ws[0];
#pragma unroll
    for (int i = 1; i < NR; ++i) gd = (lane == i) ? (unsigned long long)p.ws[i] : gd;
    gd += WS_AMAX + (size_t)(rank * NWG + bid) * 8;
    asm volatile("" : "+v"(gd));
    bool bi = false;
    if (PF_MODE == 5) { const int kp = S + ((pg - S) & 7); if (kp < S + PF_LEAD) { burst(kp); bi = true; } }
    if (bi) wait_vm<2 * PER + 2>(); else wait_vm<2 * PER>();
    BAR;
    int slot = 0;
    for (int t = 1; t <= T - S; ++t) {
      const int kp = t + S - 1 + PF_LEAD;
      const bool b0 = PF_MODE == 5 && (kp & 7) == pg && kp < T;
      const bool b1 = PF_MODE == 5 && ((kp - 1) & 7) == pg && kp - 1 < T && t > 1;
      if (b0) burst(kp);
      issue(t + S - 1, slot);
      slot = (slot + 1) & (S - 1);
      if (b0 || b1 || (bi && t <= 2)) wait_vm<2 * PER + 2>(); else wait_vm<2 * PER>();
      BAR;
    }
    wait_vm<PER>(); BAR;
    asm volatile("s_waitcnt vmcnt(0)" : "+v"(pfd) :: "memory"); BAR;
    BAR;
    const unsigned call = __builtin_amdgcn_readfirstlane(s_call[0]);
    const unsigned par = call & 1u;
    auto ra = __builtin_amdgcn_make_buffer_rsrc((void*)(p.myws + WS_AMAX), 0, 2 * 2048 * 8, 0x00020000);
    const unsigned long long t0 = __builtin_amdgcn_s_memrealtime(); unsigned spins = 0;
    if (lw < 2) {
      if (lw == 0) {
        const float m = fmaxf(fmaxf(s_wmax[0], s_wmax[1]), fmaxf(s_wmax[2], s_wmax[3]));
        const unsigned long long g = ((unsigned long long)call << 32) | __float_as_uint(m);
        if (lane < NR) __hip_atomic_store((GAS unsigned long long*)gd + par * 2048, g, __ATOMIC_RELAXED, __HIP_MEMORY_SCOPE_SYSTEM);
        STAMP(4);
      }
      float gm = 0.f; bool bad = false;
      const int vo = par * 16384 + lw * 8192 + lane * 16;
      unsigned need = 0xffu;
      for (;;) {
        u32x4 v[8];
#pragma unroll
        for (int i = 0; i < 8; ++i) if (need & (1u << i)) v[i] = __builtin_amdgcn_raw_buffer_load_b128(ra, vo + i * 1024, 0, AUX_SYS);
#pragma unroll
        for (int i = 0; i < 8; ++i)
          if ((need & (1u << i)) && v[i][1] == call && v[i][3] == call) {
            gm = fmaxf(gm, fmaxf(__uint_as_float(v[i][0]), __uint_as_float(v[i][2])));
            need &= ~(1u << i);
          }
        if (!__any(need != 0u)) break;
        if (give_up(tmo, call, t0, spins)) { bad = true; break; }
      }
      gm = wave_max(gm);
      if (lane == 0) *((volatile LAS u32x2*)s_g + lw) = (u32x2){__float_as_uint(gm), bad ? 2u : 1u};
      if (lw == 0) STAMP(5);
    } else {
      auto ro = __builtin_amdgcn_make_buffer_rsrc((void*)p.myout, 0, N * NLOC, 0x00020000);
      const int rwv = lw - 2;
      bool bad = false;
      if (EARLY && !(bid & 1)) {
        auto rs = __builtin_amdgcn_make_buffer_rsrc((void*)(p.myws + WS_STG), 0, 2 * STG_PAR, 0x00020000);
        const unsigned pm = (((call + 1u) >> 1) & 1u) ? 0xffffffffu : 0u;
        const int src = bid >> 5, stm = (bid >> 3) & 3, sks = (bid >> 1) & 1;
        const int vo = par * STG_PAR + ((bid >> 1) * 8 + rwv * 4) * 2048 + lane * 16;
        u32x4 v[8]; u32x2 gsc[2];
        for (;;) {
#pragma unroll
          for (int i = 0; i < 8; ++i) v[i] = __builtin_amdgcn_raw_buffer_load_b128(rs, vo + i * 1024, 0, AUX_SYS);
#pragma unroll
          for (int c = 0; c < 2; ++c) {
            const int sn = rank * NLOC + (rwv * 2 + c) * BN;
            const int sb = ((((sn & 1023) >> 7) * 4 + stm) << 3) + (sn >> 10) * 2 + sks;
            gsc[c] = __builtin_amdgcn_raw_buffer_load_b64(ra, par * 16384 + (src * NWG + sb) * 8, 0, AUX_SYS);
          }
          unsigned o = 0;
#pragma unroll
          for (int i = 0; i < 8; ++i) o |= tagbits(v[i], pm);
          if (__all((o & 1u) == 0u && gsc[0][1] == call && gsc[1][1] == call)) break;
          if (give_up(tmo, call, t0, spins)) { bad = true; break; }
        }
        const float inv = wait_amax(bad);
        if (!bad) {
#pragma unroll
          for (int b = 0; b < 4; ++b) {
            const float f = __uint_as_float(gsc[b >> 1][0]) * (1.0f / 32767.0f) * inv;
            unsigned d[4];
#pragma unroll
            for (int nb = 0; nb < 4; ++nb) {
              const unsigned e0 = v[2 * b + (nb >> 1)][(nb & 1) * 2], e1 = v[2 * b + (nb >> 1)][(nb & 1) * 2 + 1];
              d[nb] = q4(lo16(e0), hi16(e0), lo16(e1), hi16(e1), f);
            }
            __builtin_amdgcn_raw_buffer_store_b128(xpose(d[0], d[1], d[2], d[3]), ro,
                (bid * 16 + (lane & 15)) * NLOC + (rwv * 4 + b) * 64 + (lane >> 4) * 16, 0, AUX_SYS);
          }
        }
      } else {
        (void)wait_amax(bad);
        const int vo = bid * 8192 + rwv * 4096 + lane * 16;
        bool need = true;
        for (;;) {
          if (need) {
            u32x4 v[4];
#pragma unroll
            for (int i = 0; i < 4; ++i) v[i] = __builtin_amdgcn_raw_buffer_load_b128(ro, vo + i * 1024, 0, AUX_SYS);
            unsigned z = 0;
#pragma unroll
            for (int i = 0; i < 4; ++i)
#pragma unroll
              for (int j = 0; j < 4; ++j) { const unsigned t = v[i][j] ^ 0x80808080u; z |= (t - 0x01010101u) & ~t & 0x80808080u; }
            need = z != 0u;
          }
          if (!__any(need)) break;
          if (give_up(tmo, call, t0, spins)) break;
        }
      }
      if (lw == 2) STAMP(7);
    }
  } else {
    const int wr = wid >> 1, wc = wid & 1;
    const int g = lane >> 4, r = lane & 15, q = r >> 2, pp = lane & 3;
    {
      auto ro = __builtin_amdgcn_make_buffer_rsrc((void*)p.myout, 0, N * NLOC, 0x00020000);
      const u32x4 f = {0x80808080u, 0x80808080u, 0x80808080u, 0x80808080u};
      const int vo = bid * 8192 + wid * 2048 + lane * 16;
      __builtin_amdgcn_raw_buffer_store_b128(f, ro, vo, 0, AUX_SYS);
      __builtin_amdgcn_raw_buffer_store_b128(f, ro, vo + 1024, 0, AUX_SYS);
    }
    GAS unsigned* cnt = (GAS unsigned*)(myws + WS_CNT + bid * 64);
    const unsigned callv = __hip_atomic_load(cnt, __ATOMIC_RELAXED, __HIP_MEMORY_SCOPE_AGENT);
    if (threadIdx.x == 0) { *s_g = (u32x4){0u, 0u, 0u, 0u}; s_call[1] = 0u; }
    unsigned base[6], cur[6];
#pragma unroll
    for (int k2 = 0; k2 < 2; ++k2) base[k2] = (wr * 64 + r) * 128 + (((4 * k2 + g) ^ ((r >> 1) & 7)) * 16);
    {
      const int krow = 8 * g + q, hx = hB(krow);
#pragma unroll
      for (int nb = 0; nb < 4; ++nb) base[2 + nb] = A_BYTES + krow * 256 + (((wc * 8 + 2 * nb + (pp >> 1)) ^ hx) * 16) + (pp & 1) * 8;
    }
    f32x4 acc[4][4];
#pragma unroll
    for (int i = 0; i < 4; ++i)
#pragma unroll
      for (int j = 0; j < 4; ++j) acc[i][j] = (f32x4){0.f, 0.f, 0.f, 0.f};
    s16x8 fa[2][2][4];
    s16x4 fb[2][2][4][2];
    auto rd1 = [&](int buf, int i) __attribute__((always_inline)) {
      const int k2 = i / 12, j = i % 12;
      if (j < 8) fb[buf][k2][j >> 1][j & 1] = __builtin_amdgcn_ds_read_tr16_b64_v4i16((LAS s16x4*)(lds + cur[2 + (j >> 1)] + (32 * k2 + 4 * (j & 1)) * 256));
      else fa[buf][k2][j - 8] = *(LAS s16x8*)(lds + cur[k2] + (j - 8) * 2048);
    };
    auto mf1 = [&](int buf, int i) __attribute__((always_inline)) {
      const int k2 = i >> 4, mb = (i >> 2) & 3, nb = i & 3;
      s16x8 b8 = __builtin_shufflevector(fb[buf][k2][nb][0], fb[buf][k2][nb][1], 0, 1, 2, 3, 4, 5, 6, 7);
      MF(acc[mb][nb], b8, fa[buf][k2][mb]);
    };
    int slot = 0;
    auto advance = [&]() __attribute__((always_inline)) {
      slot = (slot + 1) & (S - 1);
      const unsigned so = slot * SLOT;
#pragma unroll
      for (int i = 0; i < 6; ++i) cur[i] = base[i] + so;
    };
    auto stage = [&](int buf) __attribute__((always_inline)) {
      BAR; SB0;
#pragma unroll
      for (int i = 0; i < 32; ++i) {
        mf1(buf ^ 1, i);
        if (i < 24) rd1(buf, i);
        if (i == 28) advance();
        if (i == 30) wait_lgkm0();
        SB0;
      }
    };
#pragma unroll
    for (int i = 0; i < 6; ++i) cur[i] = base[i];
    BAR; SB0;
    if (wid == 0) STAMP(1);
#pragma unroll
    for (int i = 0; i < 24; ++i) rd1(0, i);
    advance();
    wait_lgkm0(); SB0;
    for (int t = 1; t < T - 1; t += 2) { stage(1); stage(0); }
    stage(1);
    const unsigned call = __builtin_amdgcn_readfirstlane(callv) + 1u, par = call & 1u;
    auto rxch = __builtin_amdgcn_make_buffer_rsrc((void*)(p.myws + WS_XCH), 0, NWG * XCH_WG, 0x00020000);
    const int vos = (bid ^ 1) * XCH_WG + wid * 8192 + lane * 16;
    SB0;
#pragma unroll
    for (int i = 0; i < 32; ++i) if (((i >> 2) & 3) >= 2) mf1(1, i);
    SB0;
    float lm = 0.f, lr = 0.f;
#pragma unroll
    for (int u = 0; u < 16; ++u) {
      mf1(1, (u >> 3) * 16 + (u & 7));
      SB0;
      if (u >= 4 && u < 8) {
#pragma unroll
        for (int c = 2 * (u - 4); c < 2 * (u - 4) + 2; ++c) {
          const f32x4 a = acc[2 + (c >> 2)][c & 3];
          lm = fmaxf(fmaxf(lm, fmaxf(fabsf(a[0]), fabsf(a[1]))), fmaxf(fabsf(a[2]), fabsf(a[3])));
        }
        if (u == 7) {
          lr = lm > 0.f ? __builtin_amdgcn_rcpf(lm) : 0.f;
          __builtin_amdgcn_raw_buffer_store_b32((__float_as_uint(lm) & ~1u) | par, rxch, (bid ^ 1) * XCH_WG + wid * 8192 + 4096 + lane * 4, 0, AUX_AGENT);
        }
        SB0;
      }
      if (u >= 8 && u < 12) {
        const int c = 2 * (u - 8);
        const f32x4 a = acc[2 + (c >> 2)][c & 3] * lr, b = acc[2 + (c >> 2)][(c & 3) + 1] * lr;
        const u32x4 e = {enc16(a[0], a[1], par), enc16(a[2], a[3], par), enc16(b[0], b[1], par), enc16(b[2], b[3], par)};
        __builtin_amdgcn_raw_buffer_store_b128(e, rxch, vos + (u - 8) * 1024, 0, AUX_AGENT);
        SB0;
      }
    }
    asm volatile("s_nop 15\n\ts_nop 7" ::: "memory");
    SB0;
    if (wid == 0) STAMP(2);
    bool bad = false;
    {
      const unsigned pm = par ? 0xffffffffu : 0u;
      const int vox = bid * XCH_WG + wid * 8192 + lane * 16;
      const unsigned long long t0 = __builtin_amdgcn_s_memrealtime(); unsigned spins = 0;
      for (;;) {
        u32x4 v[4];
#pragma unroll
        for (int i = 0; i < 4; ++i) v[i] = __builtin_amdgcn_raw_buffer_load_b128(rxch, vox + i * 1024, 0, AUX_AGENT);
        const unsigned sm = __builtin_amdgcn_raw_buffer_load_b32(rxch, bid * XCH_WG + wid * 8192 + 4096 + lane * 4, 0, AUX_AGENT);
        unsigned o = sm ^ pm;
#pragma unroll
        for (int i = 0; i < 4; ++i) o |= tagbits(v[i], pm);
        if (__all((o & 1u) == 0u)) {
          const float f = __uint_as_float(sm) * (1.0f / 32767.0f);
#pragma unroll
          for (int c = 0; c < 8; ++c) {
            const unsigned e0 = v[c >> 1][(c & 1) * 2], e1 = v[c >> 1][(c & 1) * 2 + 1];
            f32x4& a = acc[c >> 2][c & 3];
            a[0] = fmaf(lo16(e0), f, a[0]); a[1] = fmaf(hi16(e0), f, a[1]); a[2] = fmaf(lo16(e1), f, a[2]); a[3] = fmaf(hi16(e1), f, a[3]);
          }
          break;
        }
        if (give_up(tmo, call, t0, spins)) { bad = true; break; }
      }
    }
    wait_vm<0>();
    float m = 0.f;
#pragma unroll
    for (int i = 0; i < 2; ++i)
#pragma unroll
      for (int nb = 0; nb < 4; ++nb)
#pragma unroll
        for (int j = 0; j < 4; ++j) m = fmaxf(m, fabsf(acc[i][nb][j]));
    m = wave_max(m);
    if (lane == 0) { s_wmax[wid] = m; if (bad) s_call[1] = 1u; if (wid == 0) s_call[0] = call; }
    wait_lgkm0();
    if (wid == 0) STAMP(3);
    BAR;
    const unsigned dest = ((unsigned)n0 / NLOC) & (NR - 1);
    bad = s_call[1] != 0u;
    if (EARLY && !bad) {
      const float tm_ = fmaxf(fmaxf(s_wmax[0], s_wmax[1]), fmaxf(s_wmax[2], s_wmax[3]));
      const float tr = tm_ > 0.f ? __builtin_amdgcn_rcpf(tm_) : 0.f;
      const unsigned tag = ((call + 1u) >> 1) & 1u;
      auto rs = __builtin_amdgcn_make_buffer_rsrc((void*)(p.ws[dest] + WS_STG), 0, 2 * STG_PAR, 0x00020000);
      const int vo = par * STG_PAR + ((rank * 16 + tm * 4 + wr * 2 + ks) * 8 + (tn & 3) * 2 + wc) * 2048 + lane * 16;
#pragma unroll
      for (int h = 0; h < 2; ++h) {
        const f32x4 a = acc[0][2 * h] * tr, b = acc[0][2 * h + 1] * tr;
        const u32x4 e = {enc16(a[0], a[1], tag), enc16(a[2], a[3], tag), enc16(b[0], b[1], tag), enc16(b[2], b[3], tag)};
        __builtin_amdgcn_raw_buffer_store_b128(e, rs, vo + h * 1024, 0, AUX_SYS);
      }
    }
    if (threadIdx.x == 0) __hip_atomic_store(cnt, call, __ATOMIC_RELAXED, __HIP_MEMORY_SCOPE_AGENT);
    auto rd = __builtin_amdgcn_make_buffer_rsrc((void*)p.out[dest], 0, N * NLOC, 0x00020000);
    const int vod = (rank * MLOC + m0 + wr * 64 + ks * 32 + r) * NLOC + (n0 & (NLOC - 1)) + wc * 64 + g * 16;
    const float inv = wait_amax(bad);
#pragma unroll
    for (int i = EARLY; i < 2; ++i) {
      unsigned d[4];
#pragma unroll
      for (int nb = 0; nb < 4; ++nb) d[nb] = q4(acc[i][nb][0], acc[i][nb][1], acc[i][nb][2], acc[i][nb][3], inv);
      u32x4 o = xpose(d[0], d[1], d[2], d[3]);
      if (bad) o = (u32x4){0x80808080u, 0x80808080u, 0x80808080u, 0x80808080u};
      __builtin_amdgcn_raw_buffer_store_b128(o, rd, vod + i * 16 * NLOC, 0, AUX_SYS);
    }
    if (wid == 0) STAMP(6);
  }
#endif
}

}

extern "C" void kernel_launch(int rank, int nranks, ncclComm_t comm,
                              hipStream_t stream,
                              void* const* d_in, const int* in_sizes, int n_in,
                              void* d_out, int out_size,
                              const int* in_dtypes, int out_dtype,
                              void* const* d_out_all,
                              void* const* d_ws_all, size_t ws_bytes) {
  (void)comm; (void)in_dtypes; (void)out_dtype; (void)d_out;
  if (nranks != NR || n_in != 2 || in_sizes[0] != MLOC * K || in_sizes[1] != K * N || out_size != N * NLOC ||
      ws_bytes < WS_NEED || (unsigned)rank >= (unsigned)NR) {
    fprintf(stderr, "kernel_launch: unexpected problem shape\n");
    return;
  }
  static int ready = 0;
  if (!ready) {
    HIP_CHECK(hipFuncSetAttribute((const void*)gemm_a2a_q8, hipFuncAttributeMaxDynamicSharedMemorySize, LDS_BYTES));
    ready = 1;
  }
  Params p;
  p.x = d_in[0]; p.w = d_in[1]; p.rank = rank; p.pad = 0;
  p.myout = (char*)d_out_all[rank]; p.myws = (char*)d_ws_all[rank];
  for (int i = 0; i < NR; ++i) { p.out[i] = (char*)d_out_all[i]; p.ws[i] = (char*)d_ws_all[i]; }
  hipLaunchKernelGGL(gemm_a2a_q8, dim3(NWG), dim3(512), LDS_BYTES, stream, p);
  HIP_CHECK(hipGetLastError());
}
```

```cpp
#include <hip/hip_runtime.h>
#include <rccl/rccl.h>
#include <cstdio>

#define HIP_CHECK(expr) do { hipError_t _e = (expr); if (_e != hipSuccess) { \
    fprintf(stderr, "HIP error %s at %s:%d\n", hipGetErrorString(_e), __FILE__, __LINE__); \
    return; } } while (0)

namespace {

typedef short s16x2 __attribute__((ext_vector_type(2)));
typedef short s16x4 __attribute__((ext_vector_type(4)));
typedef short s16x8 __attribute__((ext_vector_type(8)));
typedef float f32x4 __attribute__((ext_vector_type(4)));
typedef unsigned u32x2 __attribute__((ext_vector_type(2)));
typedef unsigned u32x4 __attribute__((ext_vector_type(4)));
#define LAS __attribute__((address_space(3)))
#define GAS __attribute__((address_space(1)))

constexpr int NR = 8;
constexpr int MLOC = 512, N = 4096, K = 4096, NLOC = 512;
constexpr int BM = 128, BN = 128, BK = 64, KS = 2;
constexpr int T = K / KS / BK;
constexpr int S = 4;
constexpr int A_BYTES = BM * BK * 2, B_BYTES = BK * BN * 2, SLOT = A_BYTES + B_BYTES;
constexpr int LDS_MISC = S * SLOT;
constexpr int LDS_BYTES = S * SLOT + 256;
constexpr int NWG = 256;
constexpr int PER = 8;

constexpr size_t WS_TMO = 0;
constexpr size_t WS_CNT = 4096;
constexpr size_t WS_LOC = 65536;
constexpr size_t WS_AMAX = 131072;
constexpr size_t WS_DBG = 262144;
constexpr size_t WS_XCH = 1u << 20;
constexpr int XCH_WG = 32768;
constexpr size_t WS_NEED = WS_XCH + (size_t)NWG * XCH_WG;
constexpr unsigned long long DEADLINE = 50000000ull;

#define PF_MODE 5
#define PF_LEAD 3
#define POLL_SLEEP 2
#define G_AUX AUX_SYS
#define G_REP 1
#define XCH16 1
#define XS 0
#define XS_LEAD 3
#define CPRIO 0
#define LEADER 0
#define XDELAY 0
#define LAXW 0
#define PF_INIT 0

struct Params {
  const void* x; const void* w;
  char* myout; char* myws;
  int rank, pad;
  char* out[NR]; char* ws[NR];
};

#define MEMFENCE asm volatile("" ::: "memory")
#define SB0 __builtin_amdgcn_sched_barrier(0)
#define BAR do { MEMFENCE; __builtin_amdgcn_s_barrier(); MEMFENCE; } while (0)
#define MF(acc, a, b) asm volatile("v_mfma_f32_16x16x32_bf16 %0, %1, %2, %0" : "+v"(acc) : "v"(a), "v"(b))
template <int N_> __device__ __forceinline__ void wait_vm() { asm volatile("s_waitcnt vmcnt(%0)" :: "i"(N_) : "memory"); }
__device__ __forceinline__ void wait_lgkm0() { MEMFENCE; __builtin_amdgcn_s_waitcnt(0xC07F); MEMFENCE; }
__device__ __forceinline__ int hB(int row) { return ((row & 3) | (((row >> 3) & 1) << 2)) << 1; }
constexpr int AUX_AGENT = 16, AUX_SYS = 17;
constexpr float MAGIC = 12582912.0f;

#define STAMP(i) do { } while (0)

__device__ __forceinline__ float wave_max(float f) {
  unsigned v = __float_as_uint(f);
  v = max(v, (unsigned)__builtin_amdgcn_update_dpp(0, (int)v, 0xB1, 0xf, 0xf, false));
  v = max(v, (unsigned)__builtin_amdgcn_update_dpp(0, (int)v, 0x4E, 0xf, 0xf, false));
  v = max(v, (unsigned)__builtin_amdgcn_update_dpp(0, (int)v, 0x141, 0xf, 0xf, false));
  v = max(v, (unsigned)__builtin_amdgcn_update_dpp(0, (int)v, 0x140, 0xf, 0xf, false));
  const unsigned a = __builtin_amdgcn_readlane(v, 0), b = __builtin_amdgcn_readlane(v, 16);
  const unsigned c = __builtin_amdgcn_readlane(v, 32), d = __builtin_amdgcn_readlane(v, 48);
  return __uint_as_float(max(max(a, b), max(c, d)));
}

__device__ __forceinline__ bool give_up(GAS unsigned* tmo, unsigned call, unsigned long long t0, unsigned& spins) {
  __builtin_amdgcn_s_sleep(POLL_SLEEP);
  if ((++spins & 15u) != 0u) return false;
  if ((unsigned)__builtin_amdgcn_readfirstlane(__hip_atomic_load(tmo, __ATOMIC_RELAXED, __HIP_MEMORY_SCOPE_AGENT)) == call) return true;
  if (__builtin_amdgcn_s_memrealtime() - t0 > DEADLINE) {
    __hip_atomic_store(tmo, call, __ATOMIC_RELAXED, __HIP_MEMORY_SCOPE_AGENT);
    return true;
  }
  return false;
}
__device__ __forceinline__ unsigned enc16(float a, float b, unsigned tag) {
  return (__builtin_bit_cast(unsigned, __builtin_amdgcn_cvt_pknorm_i16(a, b)) & ~1u) | tag;
}
__device__ __forceinline__ float lo16(unsigned v) { return (float)(short)(v & 0xffffu); }
__device__ __forceinline__ float hi16(unsigned v) { return (float)((int)v >> 16); }
__device__ __forceinline__ unsigned tagbits(const u32x4& v, unsigned pm) { return (v[0] ^ pm) | (v[1] ^ pm) | (v[2] ^ pm) | (v[3] ^ pm); }
__device__ __forceinline__ unsigned q4(float a, float b, float c, float d, float f) {
  const unsigned b0 = __float_as_uint(__builtin_amdgcn_fmed3f(fmaf(a, f, MAGIC), MAGIC - 127.0f, MAGIC + 127.0f));
  const unsigned b1 = __float_as_uint(__builtin_amdgcn_fmed3f(fmaf(b, f, MAGIC), MAGIC - 127.0f, MAGIC + 127.0f));
  const unsigned b2 = __float_as_uint(__builtin_amdgcn_fmed3f(fmaf(c, f, MAGIC), MAGIC - 127.0f, MAGIC + 127.0f));
  const unsigned b3 = __float_as_uint(__builtin_amdgcn_fmed3f(fmaf(d, f, MAGIC), MAGIC - 127.0f, MAGIC + 127.0f));
  return __builtin_amdgcn_perm(__builtin_amdgcn_perm(b3, b2, 0x0c0c0400u), __builtin_amdgcn_perm(b1, b0, 0x0c0c0400u), 0x05040100u);
}
__device__ __forceinline__ u32x4 xpose(unsigned d0, unsigned d1, unsigned d2, unsigned d3) {
  { auto t = __builtin_amdgcn_permlane32_swap(d0, d2, false, false); d0 = t[0]; d2 = t[1]; }
  { auto t = __builtin_amdgcn_permlane32_swap(d1, d3, false, false); d1 = t[0]; d3 = t[1]; }
  { auto t = __builtin_amdgcn_permlane16_swap(d0, d1, false, false); d0 = t[0]; d1 = t[1]; }
  { auto t = __builtin_amdgcn_permlane16_swap(d2, d3, false, false); d2 = t[0]; d3 = t[1]; }
  return (u32x4){d0, d1, d2, d3};
}

__global__ __launch_bounds__(512, 2) void gemm_a2a_q8(const Params p) {
#if defined(__HIP_DEVICE_COMPILE__)
  extern __shared__ __attribute__((aligned(16))) char smem[];
  LAS char* lds = (LAS char*)smem;
  LAS float* s_wmax = (LAS float*)(lds + LDS_MISC);
  LAS unsigned* s_call = (LAS unsigned*)(lds + LDS_MISC + 16);
  volatile LAS u32x4* s_g = (volatile LAS u32x4*)(lds + LDS_MISC + 32);
  LAS unsigned* s_lead = (LAS unsigned*)(lds + LDS_MISC + 64);
  const int bid = blockIdx.x;
  const int xcd = bid & 7, idx = bid >> 3;
  const int ks = xcd & 1, tm = idx & 3, tn = idx >> 2;
  const int m0 = tm * BM, n0 = (xcd >> 1) * 1024 + tn * BN;
  const int kbase = ks * (K / KS);
  const int wid = __builtin_amdgcn_readfirstlane(threadIdx.x >> 6), lane = threadIdx.x & 63;
  const int rank = p.rank;
  GAS char* myws = (GAS char*)p.myws;
  GAS unsigned* tmo = (GAS unsigned*)(myws + WS_TMO);
  auto wait_amax = [&](bool& bad) __attribute__((always_inline)) -> float {
    u32x4 g;
    for (;;) { g = *s_g; if (g[1] != 0u && g[3] != 0u) break; __builtin_amdgcn_s_sleep(1); }
    bad = bad || g[1] != 1u || g[3] != 1u;
    const float scale = fmaxf(__uint_as_float(g[0]), __uint_as_float(g[2])) / 127.0f;
    return scale > 0.f ? 1.0f / scale : 0.f;
  };

  if (wid >= 4) {
    const int lw = wid - 4;
    auto rx = __builtin_amdgcn_make_buffer_rsrc((void*)p.x, 0, MLOC * K * 2, 0x00020000);
    auto rw = __builtin_amdgcn_make_buffer_rsrc((void*)p.w, 0, K * N * 2, 0x00020000);
    int voA[4], voB[4];
#pragma unroll
    for (int i = 0; i < 4; ++i) {
      const int row = 8 * (lw * 4 + i) + (lane >> 3), ch = (lane & 7) ^ ((row >> 1) & 7);
      voA[i] = (m0 + (row ^ (ks * 32))) * (K * 2) + kbase * 2 + ch * 16;
    }
#pragma unroll
    for (int i = 0; i < 4; ++i) {
      const int krow = 4 * (lw * 4 + i) + (lane >> 4), ch = (lane & 15) ^ hB(krow);
      voB[i] = (kbase + krow) * (N * 2) + n0 * 2 + ch * 16;
    }
    auto issue = [&](int kt, int slot) __attribute__((always_inline)) {
      const int soA = kt * (BK * 2), soB = kt * (BK * N * 2);
      LAS char* sb = lds + slot * SLOT;
#pragma unroll
      for (int i = 0; i < 4; ++i) {
        __builtin_amdgcn_raw_ptr_buffer_load_lds(rx, (LAS void*)(sb + (lw * 4 + i) * 1024), 16, voA[i], soA, 0, 0);
        __builtin_amdgcn_raw_ptr_buffer_load_lds(rw, (LAS void*)(sb + A_BYTES + (lw * 4 + i) * 1024), 16, voB[i], soB, 0, 0);
      }
    };
    unsigned pfd = 0;
    const int pg = idx >> 2, pj = (idx & 3) * 4 + lw;
    const int qvw = (kbase + pj * 4 + (lane >> 4)) * (N * 2) + (xcd >> 1) * 2048 + (lane & 15) * 128;
    const int qvx = (pj * 32 + (lane & 31)) * (K * 2) + kbase * 2;
    auto burst = [&](int kt) __attribute__((always_inline)) {
      asm volatile("buffer_load_dword %0, %1, %2, %3 offen" : "+v"(pfd) : "v"(qvw), "s"(rw), "s"(kt * (BK * N * 2)));
      if (!XS) asm volatile("buffer_load_dword %0, %1, %2, %3 offen" : "+v"(pfd) : "v"(qvx), "s"(rx), "s"(kt * (BK * 2)));
    };
    constexpr int NB_ = XS ? 1 : 2;
    unsigned sfd = 0;
    const char* xs = (const char*)p.x + (size_t)(idx * 4 + lw) * 4 * (K * 2) + kbase * 2;
#define XPF(kt) do { asm volatile("s_waitcnt lgkmcnt(8)\n\ts_load_dword %0, %1, %2\n\ts_load_dword %0, %1, %3\n\ts_load_dword %0, %1, %4\n\ts_load_dword %0, %1, %5" \
      : "+s"(sfd) : "s"(xs), "i"((kt) * 128), "i"((kt) * 128 + 8192), "i"((kt) * 128 + 16384), "i"((kt) * 128 + 24576)); } while (0)
    STAMP(0);
    if (PF_INIT) {
      const int sh = idx * 4 + lw;
      const int sw = min(lane >> 3, PF_INIT - 1), sx = min((lane >> 2) & 7, PF_INIT - 1);
      const int vw = (kbase + sw * BK + (sh >> 1)) * (N * 2) + (xcd >> 1) * 2048 + ((sh & 1) * 8 + (lane & 7)) * 128;
      const int vx = (sh * 4 + (lane & 3)) * (K * 2) + (kbase + sx * BK) * 2;
      asm volatile("buffer_load_dword %0, %1, %2, 0 offen" : "+v"(pfd) : "v"(vw), "s"(rw));
      asm volatile("buffer_load_dword %0, %1, %2, 0 offen" : "+v"(pfd) : "v"(vx), "s"(rx));
    }
    issue(0, 0); issue(1, 1);
    unsigned gdl = 0, gdh = 0;
    if (bid == LEADER) {
#pragma unroll
      for (int i = 1; i < NR; ++i) {
        const bool c = (lane & 7) == i;
        unsigned lo = (unsigned)(unsigned long long)p.ws[i], hi = (unsigned)((unsigned long long)p.ws[i] >> 32);
        asm volatile("" : "+s"(lo), "+s"(hi));
        gdl = c ? lo : gdl; gdh = c ? hi : gdh;
      }
      gdl = (lane & 7) == 0 ? (unsigned)(unsigned long long)p.ws[0] : gdl;
      gdh = (lane & 7) == 0 ? (unsigned)((unsigned long long)p.ws[0] >> 32) : gdh;
    }
    unsigned long long gd = (((unsigned long long)gdh << 32) | gdl) + (WS_AMAX + (size_t)((lane >> 3) * 8 + rank) * 128);
    asm volatile("" : "+v"(gd));
    wait_vm<PER>(); BAR;
    issue(2, 2); issue(3, 3);
    bool bi = false;
    if (PF_MODE == 5) { const int kp = S + ((pg - S) & 7); if (kp < S + PF_LEAD && kp >= PF_INIT) { burst(kp); bi = true; } }
    if (XS) {
#pragma unroll
      for (int kt = S; kt < S + XS_LEAD; ++kt) XPF(kt);
    }
    if (LAXW && bi) wait_vm<2 * PER + NB_>(); else wait_vm<2 * PER>();
    BAR;
    int slot = 0;
#pragma unroll
    for (int t = 1; t <= T - S; ++t) {
      const int kp = t + S - 1 + PF_LEAD;
      const bool b0 = PF_MODE == 5 && (kp & 7) == pg && kp < T && kp >= PF_INIT;
      const bool b1 = PF_MODE == 5 && ((kp - 1) & 7) == pg && kp - 1 < T && kp - 1 >= PF_INIT && t > 1;
      if (b0) burst(kp);
      issue(t + S - 1, slot);
      slot = (slot + 1) & (S - 1);
      if (XS && t + S - 1 + XS_LEAD < T) XPF(t + S - 1 + XS_LEAD);
      if (LAXW && (b0 || b1 || (bi && t <= 2))) wait_vm<2 * PER + NB_>(); else wait_vm<2 * PER>();
      BAR;
    }
    wait_vm<PER>(); BAR;
    asm volatile("s_waitcnt vmcnt(0) lgkmcnt(0)" : "+v"(pfd), "+s"(sfd) :: "memory"); BAR;
    BAR;
    const unsigned call = __builtin_amdgcn_readfirstlane(s_call[0]);
    const unsigned par = call & 1u;
    const unsigned long long t0 = __builtin_amdgcn_s_memrealtime(); unsigned spins = 0;
    if (lw == 0) {
      const float m = fmaxf(fmaxf(s_wmax[0], s_wmax[1]), fmaxf(s_wmax[2], s_wmax[3]));
      if (lane == 0) __hip_atomic_store((GAS unsigned long long*)(myws + WS_LOC + bid * 64), ((unsigned long long)call << 32) | __float_as_uint(m),
                                        __ATOMIC_RELAXED, __HIP_MEMORY_SCOPE_AGENT);
      STAMP(4);
    }
    if (bid == LEADER) {
      GAS unsigned long long* lg = (GAS unsigned long long*)(myws + WS_LOC + (lw * 64 + lane) * 64);
      unsigned v = 0; bool need = true, bad = false;
      for (;;) {
        if (need) {
          const unsigned long long g = __hip_atomic_load(lg, __ATOMIC_RELAXED, __HIP_MEMORY_SCOPE_AGENT);
          if ((unsigned)(g >> 32) == call) { v = (unsigned)g; need = false; }
        }
        if (!__any(need)) break;
        if (give_up(tmo, call, t0, spins)) { bad = true; break; }
      }
      if (!bad) {
        const unsigned wm = __float_as_uint(wave_max(__uint_as_float(v)));
        unsigned n = 0;
        if (lane == 0) {
          __hip_atomic_fetch_max(s_lead, wm, __ATOMIC_RELAXED, __HIP_MEMORY_SCOPE_WORKGROUP);
          n = __hip_atomic_fetch_add(s_lead + 1, 1u, __ATOMIC_RELAXED, __HIP_MEMORY_SCOPE_WORKGROUP);
        }
        if (__builtin_amdgcn_readfirstlane(n) == 3u) {
          const unsigned rm = *(volatile LAS unsigned*)s_lead;
          __hip_atomic_store((GAS unsigned long long*)(gd + par * 8192), ((unsigned long long)call << 32) | rm, __ATOMIC_RELAXED, __HIP_MEMORY_SCOPE_SYSTEM);
          STAMP(10);
        }
      }
    }
    if (lw < 2) {
      if (lw == 0) {
        GAS unsigned long long* rg = (GAS unsigned long long*)(myws + WS_AMAX + par * 8192 + (xcd * 8 + (lane & 7)) * 128);
        unsigned v = 0; bool need = lane < NR, bad = false;
        for (;;) {
          if (need) {
            const unsigned long long g = __hip_atomic_load(rg, __ATOMIC_RELAXED, __HIP_MEMORY_SCOPE_SYSTEM);
            if ((unsigned)(g >> 32) == call) { v = (unsigned)g; need = false; }
          }
          if (!__any(need)) break;
          if (give_up(tmo, call, t0, spins)) { bad = true; break; }
        }
        const unsigned gm = __float_as_uint(wave_max(__uint_as_float(v)));
        if (lane == 0) *s_g = (u32x4){gm, bad ? 2u : 1u, gm, bad ? 2u : 1u};
        STAMP(5);
      }
    } else {
      auto ro = __builtin_amdgcn_make_buffer_rsrc((void*)p.myout, 0, N * NLOC, 0x00020000);
      bool bad = false;
      (void)wait_amax(bad);
      const int vo = bid * 8192 + (lw - 2) * 4096 + lane * 16;
      unsigned need = 0xfu;
      for (;;) {
        u32x4 v[4];
#pragma unroll
        for (int i = 0; i < 4; ++i) if (need & (1u << i)) v[i] = __builtin_amdgcn_raw_buffer_load_b128(ro, vo + i * 1024, 0, AUX_SYS);
#pragma unroll
        for (int i = 0; i < 4; ++i)
          if (need & (1u << i)) {
            unsigned z = 0;
#pragma unroll
            for (int j = 0; j < 4; ++j) { const unsigned t = v[i][j] ^ 0x80808080u; z |= (t - 0x01010101u) & ~t & 0x80808080u; }
            if (z == 0u) need &= ~(1u << i);
          }
        if (!__any(need != 0u)) break;
        if (give_up(tmo, call, t0, spins)) break;
      }
      if (lw == 2) STAMP(7);
    }
  } else {
    const int wr = wid >> 1, wc = wid & 1;
    __builtin_amdgcn_s_setprio(CPRIO);
    const int g = lane >> 4, r = lane & 15, q = r >> 2, pp = lane & 3;
    {
      auto ro = __builtin_amdgcn_make_buffer_rsrc((void*)p.myout, 0, N * NLOC, 0x00020000);
      const u32x4 f = {0x80808080u, 0x80808080u, 0x80808080u, 0x80808080u};
      const int vo = bid * 8192 + wid * 2048 + lane * 16;
      __builtin_amdgcn_raw_buffer_store_b128(f, ro, vo, 0, AUX_SYS);
      __builtin_amdgcn_raw_buffer_store_b128(f, ro, vo + 1024, 0, AUX_SYS);
    }
    GAS unsigned* cnt = (GAS unsigned*)(myws + WS_CNT + bid * 64);
    const unsigned callv = __hip_atomic_load(cnt, __ATOMIC_RELAXED, __HIP_MEMORY_SCOPE_AGENT);
    if (threadIdx.x == 0) { *s_g = (u32x4){0u, 0u, 0u, 0u}; s_call[1] = 0u; s_lead[0] = 0u; s_lead[1] = 0u; }
    unsigned base[6], cur[6];
#pragma unroll
    for (int k2 = 0; k2 < 2; ++k2) base[k2] = (wr * 64 + r) * 128 + (((4 * k2 + g) ^ ((r >> 1) & 7)) * 16);
    {
      const int krow = 8 * g + q, hx = hB(krow);
#pragma unroll
      for (int nb = 0; nb < 4; ++nb) base[2 + nb] = A_BYTES + krow * 256 + (((wc * 8 + 2 * nb + (pp >> 1)) ^ hx) * 16) + (pp & 1) * 8;
    }
    f32x4 acc[4][4];
#pragma unroll
    for (int i = 0; i < 4; ++i)
#pragma unroll
      for (int j = 0; j < 4; ++j) acc[i][j] = (f32x4){0.f, 0.f, 0.f, 0.f};
    s16x8 fa[2][2][4];
    s16x4 fb[2][2][4][2];
    auto rd1 = [&](int buf, int i) __attribute__((always_inline)) {
      const int k2 = i / 12, j = i % 12;
      if (j < 8) fb[buf][k2][j >> 1][j & 1] = __builtin_amdgcn_ds_read_tr16_b64_v4i16((LAS s16x4*)(lds + cur[2 + (j >> 1)] + (32 * k2 + 4 * (j & 1)) * 256));
      else fa[buf][k2][j - 8] = *(LAS s16x8*)(lds + cur[k2] + (j - 8) * 2048);
    };
    auto mf1 = [&](int buf, int i) __attribute__((always_inline)) {
      const int k2 = i >> 4, mb = (i >> 2) & 3, nb = i & 3;
      s16x8 b8 = __builtin_shufflevector(fb[buf][k2][nb][0], fb[buf][k2][nb][1], 0, 1, 2, 3, 4, 5, 6, 7);
      MF(acc[mb][nb], b8, fa[buf][k2][mb]);
    };
    int slot = 0;
    auto advance = [&]() __attribute__((always_inline)) {
      slot = (slot + 1) & (S - 1);
      const unsigned so = slot * SLOT;
#pragma unroll
      for (int i = 0; i < 6; ++i) cur[i] = base[i] + so;
    };
    auto stage = [&](int buf) __attribute__((always_inline)) {
      BAR; SB0;
#pragma unroll
      for (int i = 0; i < 32; ++i) {
        mf1(buf ^ 1, i);
        if (i < 24) rd1(buf, i);
        if (i == 28) advance();
        if (i == 30) wait_lgkm0();
        SB0;
      }
    };
#pragma unroll
    for (int i = 0; i < 6; ++i) cur[i] = base[i];
    BAR; SB0;
    if (wid == 0) STAMP(1);
#pragma unroll
    for (int i = 0; i < 24; ++i) rd1(0, i);
    advance();
    wait_lgkm0(); SB0;
    for (int t = 1; t < T - 1; t += 2) { stage(1); stage(0); }
    stage(1);
    const unsigned call = __builtin_amdgcn_readfirstlane(callv) + 1u, par = call & 1u;
    auto rxch = __builtin_amdgcn_make_buffer_rsrc((void*)(p.myws + WS_XCH), 0, NWG * XCH_WG, 0x00020000);
    const int vos = (bid ^ 1) * XCH_WG + wid * 8192 + lane * 16;
    SB0;
#pragma unroll
    for (int i = 0; i < 32; ++i) if (((i >> 2) & 3) >= 2) mf1(1, i);
    SB0;
    float lm = 0.f, lr = 0.f;
#pragma unroll
    for (int u = 0; u < 16; ++u) {
      mf1(1, (u >> 3) * 16 + (u & 7));
      SB0;
      if (u >= 4 && u < 8) {
#pragma unroll
        for (int c = 2 * (u - 4); c < 2 * (u - 4) + 2; ++c) {
          const f32x4 a = acc[2 + (c >> 2)][c & 3];
          lm = fmaxf(fmaxf(lm, fmaxf(fabsf(a[0]), fabsf(a[1]))), fmaxf(fabsf(a[2]), fabsf(a[3])));
        }
        if (u == 7) {
          lr = lm > 0.f ? __builtin_amdgcn_rcpf(lm) : 0.f;
          __builtin_amdgcn_raw_buffer_store_b32((__float_as_uint(lm) & ~1u) | par, rxch, (bid ^ 1) * XCH_WG + wid * 8192 + 4096 + lane * 4, 0, AUX_AGENT);
        }
        SB0;
      }
      if (u >= 8 && u < 12) {
        const int c = 2 * (u - 8);
        const f32x4 a = acc[2 + (c >> 2)][c & 3] * lr, b = acc[2 + (c >> 2)][(c & 3) + 1] * lr;
        const u32x4 e = {enc16(a[0], a[1], par), enc16(a[2], a[3], par), enc16(b[0], b[1], par), enc16(b[2], b[3], par)};
        __builtin_amdgcn_raw_buffer_store_b128(e, rxch, vos + (u - 8) * 1024, 0, AUX_AGENT);
        SB0;
      }
    }
    asm volatile("s_nop 15\n\ts_nop 7" ::: "memory");
    SB0;
    if (wid == 0) STAMP(2);
    bool bad = false;
    {
      const unsigned pm = par ? 0xffffffffu : 0u;
      const int vox = bid * XCH_WG + wid * 8192 + lane * 16;
      const unsigned long long t0 = __builtin_amdgcn_s_memrealtime(); unsigned spins = 0;
      for (;;) {
        u32x4 v[4];
#pragma unroll
        for (int i = 0; i < 4; ++i) v[i] = __builtin_amdgcn_raw_buffer_load_b128(rxch, vox + i * 1024, 0, AUX_AGENT);
        const unsigned sm = __builtin_amdgcn_raw_buffer_load_b32(rxch, bid * XCH_WG + wid * 8192 + 4096 + lane * 4, 0, AUX_AGENT);
        unsigned o = sm ^ pm;
#pragma unroll
        for (int i = 0; i < 4; ++i) o |= tagbits(v[i], pm);
        if (__all((o & 1u) == 0u)) {
          const float f = __uint_as_float(sm) * (1.0f / 32767.0f);
#pragma unroll
          for (int c = 0; c < 8; ++c) {
            const unsigned e0 = v[c >> 1][(c & 1) * 2], e1 = v[c >> 1][(c & 1) * 2 + 1];
            f32x4& a = acc[c >> 2][c & 3];
            a[0] = fmaf(lo16(e0), f, a[0]); a[1] = fmaf(hi16(e0), f, a[1]); a[2] = fmaf(lo16(e1), f, a[2]); a[3] = fmaf(hi16(e1), f, a[3]);
          }
          break;
        }
        if (give_up(tmo, call, t0, spins)) { bad = true; break; }
      }
    }
    wait_vm<0>();
    float m = 0.f;
#pragma unroll
    for (int i = 0; i < 2; ++i)
#pragma unroll
      for (int nb = 0; nb < 4; ++nb)
#pragma unroll
        for (int j = 0; j < 4; ++j) m = fmaxf(m, fabsf(acc[i][nb][j]));
    m = wave_max(m);
    if (lane == 0) { s_wmax[wid] = m; if (bad) s_call[1] = 1u; if (wid == 0) s_call[0] = call; }
    wait_lgkm0();
    if (wid == 0) STAMP(3);
    BAR;
    const unsigned dest = ((unsigned)n0 / NLOC) & (NR - 1);
    bad = s_call[1] != 0u;
    if (threadIdx.x == 0) __hip_atomic_store(cnt, call, __ATOMIC_RELAXED, __HIP_MEMORY_SCOPE_AGENT);
    auto rd = __builtin_amdgcn_make_buffer_rsrc((void*)p.out[dest], 0, N * NLOC, 0x00020000);
    const int vod = (rank * MLOC + m0 + wr * 64 + ks * 32 + r) * NLOC + (n0 & (NLOC - 1)) + wc * 64 + g * 16;
    const float inv = wait_amax(bad);
#pragma unroll
    for (int i = 0; i < 2; ++i) {
      unsigned d[4];
#pragma unroll
      for (int nb = 0; nb < 4; ++nb) d[nb] = q4(acc[i][nb][0], acc[i][nb][1], acc[i][nb][2], acc[i][nb][3], inv);
      u32x4 o = xpose(d[0], d[1], d[2], d[3]);
      if (bad) o = (u32x4){0x80808080u, 0x80808080u, 0x80808080u, 0x80808080u};
      __builtin_amdgcn_raw_buffer_store_b128(o, rd, vod + i * 16 * NLOC, 0, AUX_SYS);
    }
    if (wid == 0) STAMP(6);
  }
#endif
}

}

extern "C" void kernel_launch(int rank, int nranks, ncclComm_t comm,
                              hipStream_t stream,
                              void* const* d_in, const int* in_sizes, int n_in,
                              void* d_out, int out_size,
                              const int* in_dtypes, int out_dtype,
                              void* const* d_out_all,
                              void* const* d_ws_all, size_t ws_bytes) {
  (void)comm; (void)in_dtypes; (void)out_dtype; (void)d_out;
  if (nranks != NR || n_in != 2 || in_sizes[0] != MLOC * K || in_sizes[1] != K * N || out_size != N * NLOC ||
      ws_bytes < WS_NEED || (unsigned)rank >= (unsigned)NR) {
    fprintf(stderr, "kernel_launch: unexpected problem shape\n");
    return;
  }
  static int ready = 0;
  if (!ready) {
    HIP_CHECK(hipFuncSetAttribute((const void*)gemm_a2a_q8, hipFuncAttributeMaxDynamicSharedMemorySize, LDS_BYTES));
    ready = 1;
  }
  Params p;
  p.x = d_in[0]; p.w = d_in[1]; p.rank = rank; p.pad = 0;
  p.myout = (char*)d_out_all[rank]; p.myws = (char*)d_ws_all[rank];
  for (int i = 0; i < NR; ++i) { p.out[i] = (char*)d_out_all[i]; p.ws[i] = (char*)d_ws_all[i]; }
  hipLaunchKernelGGL(gemm_a2a_q8, dim3(NWG), dim3(512), LDS_BYTES, stream, p);
  HIP_CHECK(hipGetLastError());
}
```

```cpp
#include <hip/hip_runtime.h>
#include <rccl/rccl.h>
#include <cstdio>

#define HIP_CHECK(expr) do { hipError_t _e = (expr); if (_e != hipSuccess) { \
    fprintf(stderr, "HIP error %s at %s:%d\n", hipGetErrorString(_e), __FILE__, __LINE__); \
    return; } } while (0)

namespace {

typedef short s16x4 __attribute__((ext_vector_type(4)));
typedef short s16x8 __attribute__((ext_vector_type(8)));
typedef float f32x4 __attribute__((ext_vector_type(4)));
typedef unsigned u32x2 __attribute__((ext_vector_type(2)));
typedef unsigned u32x4 __attribute__((ext_vector_type(4)));
#define LAS __attribute__((address_space(3)))
#define GAS __attribute__((address_space(1)))

constexpr int NR = 8;
constexpr int MLOC = 512, N = 4096, K = 4096, NLOC = 512;
constexpr int BM = 128, BN = 128, BK = 64, KS = 2;
constexpr int T = K / KS / BK;
constexpr int S = 4;
constexpr int A_BYTES = BM * BK * 2, B_BYTES = BK * BN * 2, SLOT = A_BYTES + B_BYTES;
constexpr int LDS_MISC = S * SLOT;
constexpr int LDS_BYTES = S * SLOT + 256;
constexpr int NWG = 256;
constexpr int PER = 8;

constexpr size_t WS_TMO = 0;
constexpr size_t WS_CNT = 4096;
constexpr size_t WS_LOC = 65536;
constexpr size_t WS_AMAX = 131072;
constexpr size_t WS_DBG = 262144;
constexpr size_t WS_XCH = 1u << 20;
constexpr int XCH_WG = 32768;
constexpr size_t WS_NEED = WS_XCH + (size_t)NWG * XCH_WG;
constexpr unsigned long long DEADLINE = 50000000ull;

constexpr int PF_LEAD = 3;
constexpr int LEADER = 0;

struct Params {
  const void* x; const void* w;
  char* myout; char* myws;
  int rank, pad;
  char* out[NR]; char* ws[NR];
};

#define MEMFENCE asm volatile("" ::: "memory")
#define SB0 __builtin_amdgcn_sched_barrier(0)
#define BAR do { MEMFENCE; __builtin_amdgcn_s_barrier(); MEMFENCE; } while (0)
#define MF(acc, a, b) asm volatile("v_mfma_f32_16x16x32_bf16 %0, %1, %2, %0" : "+v"(acc) : "v"(a), "v"(b))
template <int N_> __device__ __forceinline__ void wait_vm() { asm volatile("s_waitcnt vmcnt(%0)" :: "i"(N_) : "memory"); }
__device__ __forceinline__ void wait_lgkm0() { MEMFENCE; __builtin_amdgcn_s_waitcnt(0xC07F); MEMFENCE; }
__device__ __forceinline__ int hB(int row) { return ((row & 3) | (((row >> 3) & 1) << 2)) << 1; }
constexpr int AUX_AGENT = 16, AUX_SYS = 17;
constexpr float MAGIC = 12582912.0f;

#define STAMP(i) do { } while (0)

__device__ __forceinline__ float wave_max(float f) {
  unsigned v = __float_as_uint(f);
  v = max(v, (unsigned)__builtin_amdgcn_update_dpp(0, (int)v, 0xB1, 0xf, 0xf, false));
  v = max(v, (unsigned)__builtin_amdgcn_update_dpp(0, (int)v, 0x4E, 0xf, 0xf, false));
  v = max(v, (unsigned)__builtin_amdgcn_update_dpp(0, (int)v, 0x141, 0xf, 0xf, false));
  v = max(v, (unsigned)__builtin_amdgcn_update_dpp(0, (int)v, 0x140, 0xf, 0xf, false));
  const unsigned a = __builtin_amdgcn_readlane(v, 0), b = __builtin_amdgcn_readlane(v, 16);
  const unsigned c = __builtin_amdgcn_readlane(v, 32), d = __builtin_amdgcn_readlane(v, 48);
  return __uint_as_float(max(max(a, b), max(c, d)));
}

__device__ __forceinline__ bool give_up(GAS unsigned* tmo, unsigned call, unsigned long long t0, unsigned& spins) {
  __builtin_amdgcn_s_sleep(2);
  if ((++spins & 15u) != 0u) return false;
  if ((unsigned)__builtin_amdgcn_readfirstlane(__hip_atomic_load(tmo, __ATOMIC_RELAXED, __HIP_MEMORY_SCOPE_AGENT)) == call) return true;
  if (__builtin_amdgcn_s_memrealtime() - t0 > DEADLINE) {
    __hip_atomic_store(tmo, call, __ATOMIC_RELAXED, __HIP_MEMORY_SCOPE_AGENT);
    return true;
  }
  return false;
}
__device__ __forceinline__ unsigned enc16(float a, float b, unsigned tag) {
  return (__builtin_bit_cast(unsigned, __builtin_amdgcn_cvt_pknorm_i16(a, b)) & ~1u) | tag;
}
__device__ __forceinline__ float lo16(unsigned v) { return (float)(short)(v & 0xffffu); }
__device__ __forceinline__ float hi16(unsigned v) { return (float)((int)v >> 16); }
__device__ __forceinline__ unsigned tagbits(const u32x4& v, unsigned pm) { return (v[0] ^ pm) | (v[1] ^ pm) | (v[2] ^ pm) | (v[3] ^ pm); }
__device__ __forceinline__ unsigned q4(float a, float b, float c, float d, float f) {
  const unsigned b0 = __float_as_uint(__builtin_amdgcn_fmed3f(fmaf(a, f, MAGIC), MAGIC - 127.0f, MAGIC + 127.0f));
  const unsigned b1 = __float_as_uint(__builtin_amdgcn_fmed3f(fmaf(b, f, MAGIC), MAGIC - 127.0f, MAGIC + 127.0f));
  const unsigned b2 = __float_as_uint(__builtin_amdgcn_fmed3f(fmaf(c, f, MAGIC), MAGIC - 127.0f, MAGIC + 127.0f));
  const unsigned b3 = __float_as_uint(__builtin_amdgcn_fmed3f(fmaf(d, f, MAGIC), MAGIC - 127.0f, MAGIC + 127.0f));
  return __builtin_amdgcn_perm(__builtin_amdgcn_perm(b3, b2, 0x0c0c0400u), __builtin_amdgcn_perm(b1, b0, 0x0c0c0400u), 0x05040100u);
}
__device__ __forceinline__ u32x4 xpose(unsigned d0, unsigned d1, unsigned d2, unsigned d3) {
  { auto t = __builtin_amdgcn_permlane32_swap(d0, d2, false, false); d0 = t[0]; d2 = t[1]; }
  { auto t = __builtin_amdgcn_permlane32_swap(d1, d3, false, false); d1 = t[0]; d3 = t[1]; }
  { auto t = __builtin_amdgcn_permlane16_swap(d0, d1, false, false); d0 = t[0]; d1 = t[1]; }
  { auto t = __builtin_amdgcn_permlane16_swap(d2, d3, false, false); d2 = t[0]; d3 = t[1]; }
  return (u32x4){d0, d1, d2, d3};
}

__global__ __launch_bounds__(512, 2) void gemm_a2a_q8(const Params p) {
#if defined(__HIP_DEVICE_COMPILE__)
  extern __shared__ __attribute__((aligned(16))) char smem[];
  LAS char* lds = (LAS char*)smem;
  LAS unsigned* s_call = (LAS unsigned*)(lds + LDS_MISC);
  volatile LAS u32x2* s_g = (volatile LAS u32x2*)(lds + LDS_MISC + 16);
  LAS unsigned* s_lead = (LAS unsigned*)(lds + LDS_MISC + 32);
  const int bid = blockIdx.x;
  const int xcd = bid & 7, idx = bid >> 3;
  const int ks = xcd & 1, tm = idx & 3, tn = idx >> 2;
  const int m0 = tm * BM, n0 = (xcd >> 1) * 1024 + tn * BN;
  const int kbase = ks * (K / KS);
  const int wid = __builtin_amdgcn_readfirstlane(threadIdx.x >> 6), lane = threadIdx.x & 63;
  const int rank = p.rank;
  GAS char* myws = (GAS char*)p.myws;
  GAS unsigned* tmo = (GAS unsigned*)(myws + WS_TMO);
  auto wait_amax = [&](bool& bad) __attribute__((always_inline)) -> float {
    u32x2 g;
    do { g = *s_g; } while (g[1] == 0u);
    bad = bad || g[1] != 1u;
    return __uint_as_float(g[0]);
  };

  if (wid >= 4) {
    const int lw = wid - 4;
    auto rx = __builtin_amdgcn_make_buffer_rsrc((void*)p.x, 0, MLOC * K * 2, 0x00020000);
    auto rw = __builtin_amdgcn_make_buffer_rsrc((void*)p.w, 0, K * N * 2, 0x00020000);
    int voA[4], voB[4];
#pragma unroll
    for (int i = 0; i < 4; ++i) {
      const int row = 8 * (lw * 4 + i) + (lane >> 3), ch = (lane & 7) ^ ((row >> 1) & 7);
      voA[i] = (m0 + (row ^ (ks * 32))) * (K * 2) + kbase * 2 + ch * 16;
    }
#pragma unroll
    for (int i = 0; i < 4; ++i) {
      const int krow = 4 * (lw * 4 + i) + (lane >> 4), ch = (lane & 15) ^ hB(krow);
      voB[i] = (kbase + krow) * (N * 2) + n0 * 2 + ch * 16;
    }
    auto issue = [&](int kt, int slot) __attribute__((always_inline)) {
      const int soA = kt * (BK * 2), soB = kt * (BK * N * 2);
      LAS char* sb = lds + slot * SLOT;
#pragma unroll
      for (int i = 0; i < 4; ++i) {
        __builtin_amdgcn_raw_ptr_buffer_load_lds(rx, (LAS void*)(sb + (lw * 4 + i) * 1024), 16, voA[i], soA, 0, 0);
        __builtin_amdgcn_raw_ptr_buffer_load_lds(rw, (LAS void*)(sb + A_BYTES + (lw * 4 + i) * 1024), 16, voB[i], soB, 0, 0);
      }
    };
    unsigned pfd = 0;
    const int pg = idx >> 2, pj = (idx & 3) * 4 + lw;
    const int qvw = (kbase + pj * 4 + (lane >> 4)) * (N * 2) + (xcd >> 1) * 2048 + (lane & 15) * 128;
    const int qvx = (pj * 32 + (lane & 31)) * (K * 2) + kbase * 2;
    auto burst = [&](int kt) __attribute__((always_inline)) {
      asm volatile("buffer_load_dword %0, %1, %2, %3 offen" : "+v"(pfd) : "v"(qvw), "s"(rw), "s"(kt * (BK * N * 2)));
      asm volatile("buffer_load_dword %0, %1, %2, %3 offen" : "+v"(pfd) : "v"(qvx), "s"(rx), "s"(kt * (BK * 2)));
    };
    STAMP(0);
    const int rot = (tm + tn) & 3;
    issue(rot, 0); issue((rot + 1) & 3, 1);
    unsigned gdl = 0, gdh = 0;
    if (bid == LEADER) {
#pragma unroll
      for (int i = 1; i < NR; ++i) {
        const bool c = (lane & 7) == i;
        unsigned lo = (unsigned)(unsigned long long)p.ws[i], hi = (unsigned)((unsigned long long)p.ws[i] >> 32);
        asm volatile("" : "+s"(lo), "+s"(hi));
        gdl = c ? lo : gdl; gdh = c ? hi : gdh;
      }
      gdl = (lane & 7) == 0 ? (unsigned)(unsigned long long)p.ws[0] : gdl;
      gdh = (lane & 7) == 0 ? (unsigned)((unsigned long long)p.ws[0] >> 32) : gdh;
    }
    unsigned long long gd = (((unsigned long long)gdh << 32) | gdl) + (WS_AMAX + (size_t)((lane >> 3) * 8 + rank) * 128);
    asm volatile("" : "+v"(gd));
    wait_vm<PER>(); BAR;
    issue((rot + 2) & 3, 2); issue((rot + 3) & 3, 3);
    { const int kp = S + ((pg - S) & 7); if (kp < S + PF_LEAD) burst(kp); }
    wait_vm<2 * PER>(); BAR;
#pragma unroll
    for (int t = 1; t <= T - S; ++t) {
      const int kp = t + S - 1 + PF_LEAD;
      if (kp < T && (kp & 7) == pg) burst(kp);
      issue(t + S - 1, (t - 1) & (S - 1));
      wait_vm<2 * PER>(); BAR;
    }
    wait_vm<PER>(); BAR;
    asm volatile("s_waitcnt vmcnt(0)" : "+v"(pfd) :: "memory"); BAR;
    BAR;
    const unsigned call = __builtin_amdgcn_readfirstlane(s_call[0]);
    const unsigned par = call & 1u;
    const unsigned long long t0 = __builtin_amdgcn_s_memrealtime(); unsigned spins = 0;
    if (bid == LEADER) {
      auto rl = __builtin_amdgcn_make_buffer_rsrc((void*)(p.myws + WS_LOC), 0, NWG * 64, 0x00020000);
      unsigned v = 0; bool need = true, bad = false;
      for (;;) {
        if (need) {
          const u32x4 g0 = __builtin_amdgcn_raw_buffer_load_b128(rl, (lw * 64 + lane) * 64, 0, AUX_AGENT);
          const u32x4 g1 = __builtin_amdgcn_raw_buffer_load_b128(rl, (lw * 64 + lane) * 64 + 16, 0, AUX_AGENT);
          if (g0[1] == call && g0[3] == call && g1[1] == call && g1[3] == call) { v = max(max(g0[0], g0[2]), max(g1[0], g1[2])); need = false; }
        }
        if (!__any(need)) break;
        if (give_up(tmo, call, t0, spins)) { bad = true; break; }
      }
      if (!bad) {
        const unsigned wm = __float_as_uint(wave_max(__uint_as_float(v)));
        unsigned n = 0;
        if (lane == 0) {
          __hip_atomic_fetch_max(s_lead, wm, __ATOMIC_RELAXED, __HIP_MEMORY_SCOPE_WORKGROUP);
          n = __hip_atomic_fetch_add(s_lead + 1, 1u, __ATOMIC_RELAXED, __HIP_MEMORY_SCOPE_WORKGROUP);
        }
        if (__builtin_amdgcn_readfirstlane(n) == 3u) {
          const unsigned rm = *(volatile LAS unsigned*)s_lead;
          __hip_atomic_store((GAS unsigned long long*)(gd + par * 8192), ((unsigned long long)call << 32) | rm, __ATOMIC_RELAXED, __HIP_MEMORY_SCOPE_SYSTEM);
          STAMP(10);
        }
      }
    }
    if (lw == 0) {
      GAS unsigned long long* rg = (GAS unsigned long long*)(myws + WS_AMAX + par * 8192 + (xcd * 8 + (lane & 7)) * 128);
      unsigned v = 0; bool need = lane < NR, bad = false;
      for (;;) {
        if (need) {
          const unsigned long long g = __hip_atomic_load(rg, __ATOMIC_RELAXED, __HIP_MEMORY_SCOPE_SYSTEM);
          if ((unsigned)(g >> 32) == call) { v = (unsigned)g; need = false; }
        }
        if (!__any(need)) break;
        if (give_up(tmo, call, t0, spins)) { bad = true; break; }
      }
      const float amax = wave_max(__uint_as_float(v));
      float inv;
      if (amax > 1e-30f && amax < 1e30f) {
        float rc = __builtin_amdgcn_rcpf(amax);
        rc = fmaf(fmaf(-amax, rc, 1.0f), rc, rc);
        inv = 127.0f * rc;
      } else {
        const float scale = amax / 127.0f;
        inv = scale > 0.f ? 1.0f / scale : 0.f;
      }
      if (lane == 0) *s_g = (u32x2){__float_as_uint(inv), bad ? 2u : 1u};
      STAMP(5);
    } else if (lw >= 2) {
      auto ro = __builtin_amdgcn_make_buffer_rsrc((void*)p.myout, 0, N * NLOC, 0x00020000);
      while ((*s_g)[1] == 0u) __builtin_amdgcn_s_sleep(8);
      const int vo = bid * 8192 + (lw - 2) * 4096 + lane * 16;
      unsigned need = 0xfu;
      for (;;) {
        u32x4 v[4];
#pragma unroll
        for (int i = 0; i < 4; ++i) if (need & (1u << i)) v[i] = __builtin_amdgcn_raw_buffer_load_b128(ro, vo + i * 1024, 0, AUX_SYS);
#pragma unroll
        for (int i = 0; i < 4; ++i)
          if (need & (1u << i)) {
            unsigned z = 0;
#pragma unroll
            for (int j = 0; j < 4; ++j) { const unsigned t = v[i][j] ^ 0x80808080u; z |= (t - 0x01010101u) & ~t & 0x80808080u; }
            if (z == 0u) need &= ~(1u << i);
          }
        if (!__any(need != 0u)) break;
        if (give_up(tmo, call, t0, spins)) break;
      }
      if (lw == 2) STAMP(7);
    }
  } else {
    const int wr = wid >> 1, wc = wid & 1;
    const int g = lane >> 4, r = lane & 15, q = r >> 2, pp = lane & 3;
    {
      auto ro = __builtin_amdgcn_make_buffer_rsrc((void*)p.myout, 0, N * NLOC, 0x00020000);
      const u32x4 f = {0x80808080u, 0x80808080u, 0x80808080u, 0x80808080u};
      const int vo = bid * 8192 + wid * 2048 + lane * 16;
      __builtin_amdgcn_raw_buffer_store_b128(f, ro, vo, 0, AUX_SYS);
      __builtin_amdgcn_raw_buffer_store_b128(f, ro, vo + 1024, 0, AUX_SYS);
    }
    GAS unsigned* cnt = (GAS unsigned*)(myws + WS_CNT + bid * 64);
    const unsigned callv = __hip_atomic_load(cnt, __ATOMIC_RELAXED, __HIP_MEMORY_SCOPE_AGENT);
    if (threadIdx.x == 0) { *s_g = (u32x2){0u, 0u}; s_lead[0] = 0u; s_lead[1] = 0u; }
    unsigned base[6], cur[6];
#pragma unroll
    for (int k2 = 0; k2 < 2; ++k2) base[k2] = (wr * 64 + r) * 128 + (((4 * k2 + g) ^ ((r >> 1) & 7)) * 16);
    {
      const int krow = 8 * g + q, hx = hB(krow);
#pragma unroll
      for (int nb = 0; nb < 4; ++nb) base[2 + nb] = A_BYTES + krow * 256 + (((wc * 8 + 2 * nb + (pp >> 1)) ^ hx) * 16) + (pp & 1) * 8;
    }
    f32x4 acc[4][4];
#pragma unroll
    for (int i = 0; i < 4; ++i)
#pragma unroll
      for (int j = 0; j < 4; ++j) acc[i][j] = (f32x4){0.f, 0.f, 0.f, 0.f};
    s16x8 fa[2][2][4];
    s16x4 fb[2][2][4][2];
    auto rd1 = [&](int buf, int i) __attribute__((always_inline)) {
      const int k2 = i / 12, j = i % 12;
      if (j < 8) fb[buf][k2][j >> 1][j & 1] = __builtin_amdgcn_ds_read_tr16_b64_v4i16((LAS s16x4*)(lds + cur[2 + (j >> 1)] + (32 * k2 + 4 * (j & 1)) * 256));
      else fa[buf][k2][j - 8] = *(LAS s16x8*)(lds + cur[k2] + (j - 8) * 2048);
    };
    auto mf1 = [&](int buf, int i) __attribute__((always_inline)) {
      const int k2 = i >> 4, mb = (i >> 2) & 3, nb = i & 3;
      s16x8 b8 = __builtin_shufflevector(fb[buf][k2][nb][0], fb[buf][k2][nb][1], 0, 1, 2, 3, 4, 5, 6, 7);
      MF(acc[mb][nb], b8, fa[buf][k2][mb]);
    };
    int slot = 0;
    auto advance = [&]() __attribute__((always_inline)) {
      slot = (slot + 1) & (S - 1);
      const unsigned so = slot * SLOT;
#pragma unroll
      for (int i = 0; i < 6; ++i) cur[i] = base[i] + so;
    };
    auto stage = [&](int buf) __attribute__((always_inline)) {
      BAR; SB0;
#pragma unroll
      for (int i = 0; i < 32; ++i) {
        mf1(buf ^ 1, i);
        if (i < 24) rd1(buf, i);
        if (i == 28) advance();
        if (i == 30) wait_lgkm0();
        SB0;
      }
    };
#pragma unroll
    for (int i = 0; i < 6; ++i) cur[i] = base[i];
    BAR; SB0;
    if (wid == 0) STAMP(1);
#pragma unroll
    for (int i = 0; i < 24; ++i) rd1(0, i);
    advance();
    wait_lgkm0(); SB0;
    for (int t = 1; t < T - 5; t += 2) { stage(1); stage(0); }
    stage(1);
    wait_vm<0>();
    const unsigned call = __builtin_amdgcn_readfirstlane(callv) + 1u, par = call & 1u;
    if (threadIdx.x == 0) s_call[0] = call;
    auto rxch = __builtin_amdgcn_make_buffer_rsrc((void*)(p.myws + WS_XCH), 0, NWG * XCH_WG, 0x00020000);
    const int vos = (bid ^ 1) * XCH_WG + wid * 8192 + lane * 16;
    float lm = 0.f, lr = 0.f;
    auto send = [&](int c) __attribute__((always_inline)) {
      if (c < 4) {
#pragma unroll
        for (int e = 2 * c; e < 2 * c + 2; ++e) {
          const f32x4 a = acc[2 + (e >> 2)][e & 3];
          lm = fmaxf(fmaxf(lm, fmaxf(fabsf(a[0]), fabsf(a[1]))), fmaxf(fabsf(a[2]), fabsf(a[3])));
        }
        if (c == 3) {
          lr = lm > 0.f ? __builtin_amdgcn_rcpf(lm) : 0.f;
          __builtin_amdgcn_raw_buffer_store_b32((__float_as_uint(lm) & ~1u) | par, rxch, (bid ^ 1) * XCH_WG + wid * 8192 + 4096 + lane * 4, 0, AUX_AGENT);
        }
      } else {
        const int e = 2 * (c - 4);
        const f32x4 a = acc[2 + (e >> 2)][e & 3] * lr, b = acc[2 + (e >> 2)][(e & 3) + 1] * lr;
        const u32x4 v = {enc16(a[0], a[1], par), enc16(a[2], a[3], par), enc16(b[0], b[1], par), enc16(b[2], b[3], par)};
        __builtin_amdgcn_raw_buffer_store_b128(v, rxch, vos + (c - 4) * 1024, 0, AUX_AGENT);
      }
    };
    s16x8 ta[2][2][2];
    s16x4 tb[2][2][4][2];
    auto trd = [&](int u, int j) __attribute__((always_inline)) {
      const int mb0 = u < 4 ? 2 : 0, so = (u & 3) * SLOT, k2 = j / 10, e = j % 10;
      if (e < 8) tb[u & 1][k2][e >> 1][e & 1] = __builtin_amdgcn_ds_read_tr16_b64_v4i16((LAS s16x4*)(lds + (base[2 + (e >> 1)] + so) + (32 * k2 + 4 * (e & 1)) * 256));
      else ta[u & 1][k2][e - 8] = *(LAS s16x8*)(lds + (base[k2] + so) + (mb0 + e - 8) * 2048);
    };
    auto tmf = [&](int u, int i) __attribute__((always_inline)) {
      const int mb0 = u < 4 ? 2 : 0, k2 = i >> 3, mb = (i >> 2) & 1, nb = i & 3;
      s16x8 b8 = __builtin_shufflevector(tb[u & 1][k2][nb][0], tb[u & 1][k2][nb][1], 0, 1, 2, 3, 4, 5, 6, 7);
      MF(acc[mb0 + mb][nb], b8, ta[u & 1][k2][mb]);
    };
    const int vox = bid * XCH_WG + wid * 8192 + lane * 16, voxs = bid * XCH_WG + wid * 8192 + 4096 + lane * 4;
    static_assert((T - 4) % S == 0, "tail slot mapping");
    BAR; SB0;
#pragma unroll
    for (int i = 0; i < 32; ++i) {
      mf1(1, i);
      if (i < 20) trd(0, i);
      if (i == 30) wait_lgkm0();
      SB0;
    }
#pragma unroll
    for (int u = 0; u < 8; ++u) {
      if (u < 3) { BAR; SB0; }
#pragma unroll
      for (int i = 0; i < 16; ++i) {
        tmf(u, i);
        if (u < 7 && i < 10) { trd(u + 1, 2 * i); trd(u + 1, 2 * i + 1); }
        SB0;
        if (u == 4 && i >= 6 && i < 14) { send(i - 6); SB0; }
        if (i == 14) { wait_lgkm0(); SB0; }
      }
    }
    asm volatile("s_nop 15\n\ts_nop 7" ::: "memory");
    SB0;
    if (wid == 0) STAMP(2);
    bool bad = false;
    {
      const unsigned pm = par ? 0xffffffffu : 0u;
      const unsigned long long t0 = __builtin_amdgcn_s_memrealtime(); unsigned spins = 0;
      u32x4 xv[4]; unsigned xs;
      for (;;) {
#pragma unroll
        for (int c = 0; c < 4; ++c) xv[c] = __builtin_amdgcn_raw_buffer_load_b128(rxch, vox + c * 1024, 0, AUX_AGENT);
        xs = __builtin_amdgcn_raw_buffer_load_b32(rxch, voxs, 0, AUX_AGENT);
        const unsigned o = (xs ^ pm) | tagbits(xv[0], pm) | tagbits(xv[1], pm) | tagbits(xv[2], pm) | tagbits(xv[3], pm);
        if (__all((o & 1u) == 0u)) break;
        if (give_up(tmo, call, t0, spins)) { bad = true; break; }
      }
      const float f = __uint_as_float(xs) * (1.0f / 32767.0f);
#pragma unroll
      for (int c = 0; c < 8; ++c) {
        const unsigned e0 = xv[c >> 1][(c & 1) * 2], e1 = xv[c >> 1][(c & 1) * 2 + 1];
        f32x4& a = acc[c >> 2][c & 3];
        a[0] = fmaf(lo16(e0), f, a[0]); a[1] = fmaf(hi16(e0), f, a[1]); a[2] = fmaf(lo16(e1), f, a[2]); a[3] = fmaf(hi16(e1), f, a[3]);
      }
    }
    float m = 0.f;
#pragma unroll
    for (int i = 0; i < 2; ++i)
#pragma unroll
      for (int nb = 0; nb < 4; ++nb)
#pragma unroll
        for (int j = 0; j < 4; ++j) m = fmaxf(m, fabsf(acc[i][nb][j]));
    m = wave_max(m);
    if (lane == 0) __hip_atomic_store((GAS unsigned long long*)(myws + WS_LOC + bid * 64 + wid * 8), ((unsigned long long)call << 32) | __float_as_uint(m),
                                      __ATOMIC_RELAXED, __HIP_MEMORY_SCOPE_AGENT);
    if (wid == 0) { STAMP(3); STAMP(4); }
    BAR;
    const unsigned dest = ((unsigned)n0 / NLOC) & (NR - 1);
    if (threadIdx.x == 0) __hip_atomic_store(cnt, call, __ATOMIC_RELAXED, __HIP_MEMORY_SCOPE_AGENT);
    auto rd = __builtin_amdgcn_make_buffer_rsrc((void*)p.out[dest], 0, N * NLOC, 0x00020000);
    const int vod = (rank * MLOC + m0 + wr * 64 + ks * 32 + r) * NLOC + (n0 & (NLOC - 1)) + wc * 64 + g * 16;
    const float inv = wait_amax(bad);
#pragma unroll
    for (int i = 0; i < 2; ++i) {
      unsigned d[4];
#pragma unroll
      for (int nb = 0; nb < 4; ++nb) d[nb] = q4(acc[i][nb][0], acc[i][nb][1], acc[i][nb][2], acc[i][nb][3], inv);
      u32x4 o = xpose(d[0], d[1], d[2], d[3]);
      if (bad) o = (u32x4){0x80808080u, 0x80808080u, 0x80808080u, 0x80808080u};
      __builtin_amdgcn_raw_buffer_store_b128(o, rd, vod + i * 16 * NLOC, 0, AUX_SYS);
    }
    if (wid == 0) STAMP(6);
  }
#endif
}

}

extern "C" void kernel_launch(int rank, int nranks, ncclComm_t comm,
                              hipStream_t stream,
                              void* const* d_in, const int* in_sizes, int n_in,
                              void* d_out, int out_size,
                              const int* in_dtypes, int out_dtype,
                              void* const* d_out_all,
                              void* const* d_ws_all, size_t ws_bytes) {
  (void)comm; (void)in_dtypes; (void)out_dtype; (void)d_out;
  if (nranks != NR || n_in != 2 || in_sizes[0] != MLOC * K || in_sizes[1] != K * N || out_size != N * NLOC ||
      ws_bytes < WS_NEED || (unsigned)rank >= (unsigned)NR) {
    fprintf(stderr, "kernel_launch: unexpected problem shape\n");
    return;
  }
  static int ready = 0;
  if (!ready) {
    HIP_CHECK(hipFuncSetAttribute((const void*)gemm_a2a_q8, hipFuncAttributeMaxDynamicSharedMemorySize, LDS_BYTES));
    int dev = 0, cus = 0, per_cu = 0;
    HIP_CHECK(hipGetDevice(&dev));
    HIP_CHECK(hipDeviceGetAttribute(&cus, hipDeviceAttributeMultiprocessorCount, dev));
    HIP_CHECK(hipOccupancyMaxActiveBlocksPerMultiprocessor(&per_cu, (const void*)gemm_a2a_q8, 512, LDS_BYTES));
    if (cus * per_cu < NWG) {
      fprintf(stderr, "kernel_launch: %d workgroups cannot be co-resident (%d CUs x %d)\n", NWG, cus, per_cu);
      return;
    }
    ready = 1;
  }
  Params p;
  p.x = d_in[0]; p.w = d_in[1]; p.rank = rank; p.pad = 0;
  p.myout = (char*)d_out_all[rank]; p.myws = (char*)d_ws_all[rank];
  for (int i = 0; i < NR; ++i) { p.out[i] = (char*)d_out_all[i]; p.ws[i] = (char*)d_ws_all[i]; }
  hipLaunchKernelGGL(gemm_a2a_q8, dim3(NWG), dim3(512), LDS_BYTES, stream, p);
  HIP_CHECK(hipGetLastError());
}
```

```cpp
#include <hip/hip_runtime.h>
#include <rccl/rccl.h>
#include <cstdio>

#define HIP_CHECK(expr) do { hipError_t _e = (expr); if (_e != hipSuccess) { \
    fprintf(stderr, "HIP error %s at %s:%d\n", hipGetErrorString(_e), __FILE__, __LINE__); \
    return; } } while (0)

namespace {

typedef short s16x4 __attribute__((ext_vector_type(4)));
typedef short s16x8 __attribute__((ext_vector_type(8)));
typedef float f32x4 __attribute__((ext_vector_type(4)));
typedef unsigned u32x2 __attribute__((ext_vector_type(2)));
typedef unsigned u32x4 __attribute__((ext_vector_type(4)));
#define LAS __attribute__((address_space(3)))
#define GAS __attribute__((address_space(1)))

constexpr int NR = 8;
constexpr int MLOC = 512, N = 4096, K = 4096, NLOC = 512;
constexpr int BM = 128, BN = 128, BK = 64, KS = 2;
constexpr int T = K / KS / BK;
constexpr int S = 4;
constexpr int A_BYTES = BM * BK * 2, B_BYTES = BK * BN * 2, SLOT = A_BYTES + B_BYTES;
constexpr int LDS_MISC = S * SLOT;
constexpr int LDS_BYTES = S * SLOT + 256;
constexpr int NWG = 256;
constexpr int PER = 8;

constexpr size_t WS_TMO = 0;
constexpr size_t WS_CNT = 4096;
constexpr size_t WS_LOC = 65536;
constexpr size_t WS_AMAX = 131072;
constexpr size_t WS_DBG = 262144;
constexpr size_t WS_XCH = 1u << 20;
constexpr int XCH_WG = 32768;
constexpr size_t WS_NEED = WS_XCH + (size_t)NWG * XCH_WG;
constexpr unsigned long long DEADLINE = 50000000ull;

constexpr int PF_LEAD = 3;
constexpr int LEADER = 0;

struct Params {
  const void* x; const void* w;
  char* myout; char* myws;
  int rank, pad;
  char* out[NR]; char* ws[NR];
};

#define MEMFENCE asm volatile("" ::: "memory")
#define SB0 __builtin_amdgcn_sched_barrier(0)
#define BAR do { MEMFENCE; __builtin_amdgcn_s_barrier(); MEMFENCE; } while (0)
#define MF(acc, a, b) asm volatile("v_mfma_f32_16x16x32_bf16 %0, %1, %2, %0" : "+v"(acc) : "v"(a), "v"(b))
template <int N_> __device__ __forceinline__ void wait_vm() { asm volatile("s_waitcnt vmcnt(%0)" :: "i"(N_) : "memory"); }
__device__ __forceinline__ void wait_lgkm0() { MEMFENCE; __builtin_amdgcn_s_waitcnt(0xC07F); MEMFENCE; }
__device__ __forceinline__ int hB(int row) { return ((row & 3) | (((row >> 3) & 1) << 2)) << 1; }
constexpr int AUX_AGENT = 16, AUX_SYS = 17;
constexpr float MAGIC = 12582912.0f;

#define STAMP(i) do { } while (0)

__device__ __forceinline__ float wave_max(float f) {
  unsigned v = __float_as_uint(f);
  v = max(v, (unsigned)__builtin_amdgcn_update_dpp(0, (int)v, 0xB1, 0xf, 0xf, false));
  v = max(v, (unsigned)__builtin_amdgcn_update_dpp(0, (int)v, 0x4E, 0xf, 0xf, false));
  v = max(v, (unsigned)__builtin_amdgcn_update_dpp(0, (int)v, 0x141, 0xf, 0xf, false));
  v = max(v, (unsigned)__builtin_amdgcn_update_dpp(0, (int)v, 0x140, 0xf, 0xf, false));
  const unsigned a = __builtin_amdgcn_readlane(v, 0), b = __builtin_amdgcn_readlane(v, 16);
  const unsigned c = __builtin_amdgcn_readlane(v, 32), d = __builtin_amdgcn_readlane(v, 48);
  return __uint_as_float(max(max(a, b), max(c, d)));
}

__device__ __forceinline__ bool give_up(GAS unsigned* tmo, unsigned call, unsigned long long t0, unsigned& spins) {
  __builtin_amdgcn_s_sleep(2);
  if ((++spins & 15u) != 0u) return false;
  if ((unsigned)__builtin_amdgcn_readfirstlane(__hip_atomic_load(tmo, __ATOMIC_RELAXED, __HIP_MEMORY_SCOPE_AGENT)) == call) return true;
  if (__builtin_amdgcn_s_memrealtime() - t0 > DEADLINE) {
    __hip_atomic_store(tmo, call, __ATOMIC_RELAXED, __HIP_MEMORY_SCOPE_AGENT);
    return true;
  }
  return false;
}
__device__ __forceinline__ unsigned enc16(float a, float b, unsigned tag) {
  return (__builtin_bit_cast(unsigned, __builtin_amdgcn_cvt_pknorm_i16(a, b)) & ~1u) | tag;
}
__device__ __forceinline__ float lo16(unsigned v) { return (float)(short)(v & 0xfffeu); }
__device__ __forceinline__ float hi16(unsigned v) { return (float)((int)v >> 16); }
__device__ __forceinline__ unsigned tagbits(const u32x4& v, unsigned pm) { return (v[0] ^ pm) | (v[1] ^ pm) | (v[2] ^ pm) | (v[3] ^ pm); }
__device__ __forceinline__ unsigned q4(float a, float b, float c, float d, float f) {
  const unsigned b0 = __float_as_uint(__builtin_amdgcn_fmed3f(fmaf(a, f, MAGIC), MAGIC - 127.0f, MAGIC + 127.0f));
  const unsigned b1 = __float_as_uint(__builtin_amdgcn_fmed3f(fmaf(b, f, MAGIC), MAGIC - 127.0f, MAGIC + 127.0f));
  const unsigned b2 = __float_as_uint(__builtin_amdgcn_fmed3f(fmaf(c, f, MAGIC), MAGIC - 127.0f, MAGIC + 127.0f));
  const unsigned b3 = __float_as_uint(__builtin_amdgcn_fmed3f(fmaf(d, f, MAGIC), MAGIC - 127.0f, MAGIC + 127.0f));
  return __builtin_amdgcn_perm(__builtin_amdgcn_perm(b3, b2, 0x0c0c0400u), __builtin_amdgcn_perm(b1, b0, 0x0c0c0400u), 0x05040100u);
}
__device__ __forceinline__ u32x4 xpose(unsigned d0, unsigned d1, unsigned d2, unsigned d3) {
  { auto t = __builtin_amdgcn_permlane32_swap(d0, d2, false, false); d0 = t[0]; d2 = t[1]; }
  { auto t = __builtin_amdgcn_permlane32_swap(d1, d3, false, false); d1 = t[0]; d3 = t[1]; }
  { auto t = __builtin_amdgcn_permlane16_swap(d0, d1, false, false); d0 = t[0]; d1 = t[1]; }
  { auto t = __builtin_amdgcn_permlane16_swap(d2, d3, false, false); d2 = t[0]; d3 = t[1]; }
  return (u32x4){d0, d1, d2, d3};
}

__global__ __launch_bounds__(512, 2) void gemm_a2a_q8(const Params p) {
#if defined(__HIP_DEVICE_COMPILE__)
  extern __shared__ __attribute__((aligned(16))) char smem[];
  LAS char* lds = (LAS char*)smem;
  LAS unsigned* s_call = (LAS unsigned*)(lds + LDS_MISC);
  volatile LAS u32x2* s_g = (volatile LAS u32x2*)(lds + LDS_MISC + 16);
  LAS unsigned* s_lead = (LAS unsigned*)(lds + LDS_MISC + 32);
  const int bid = blockIdx.x;
  const int xcd = bid & 7, idx = bid >> 3;
  const int ks = xcd & 1, tm = idx & 3, tn = idx >> 2;
  const int m0 = tm * BM, n0 = tn * NLOC + (((xcd >> 1) + tn) & 3) * BN;
  const int kbase = ks * (K / KS);
  const int wid = __builtin_amdgcn_readfirstlane(threadIdx.x >> 6), lane = threadIdx.x & 63;
  const int rank = p.rank;
  GAS char* myws = (GAS char*)p.myws;
  GAS unsigned* tmo = (GAS unsigned*)(myws + WS_TMO);
  auto wait_amax = [&](bool& bad) __attribute__((always_inline)) -> float {
    u32x2 g;
    do { g = *s_g; } while (g[1] == 0u);
    bad = bad || g[1] != 1u;
    return __uint_as_float(g[0]);
  };

  if (wid >= 4) {
    const int lw = wid - 4;
    auto rx = __builtin_amdgcn_make_buffer_rsrc((void*)p.x, 0, MLOC * K * 2, 0x00020000);
    auto rw = __builtin_amdgcn_make_buffer_rsrc((void*)p.w, 0, K * N * 2, 0x00020000);
    int voA[4], voB[4];
#pragma unroll
    for (int i = 0; i < 4; ++i) {
      const int row = 8 * (lw * 4 + i) + (lane >> 3), ch = (lane & 7) ^ ((row >> 1) & 7);
      voA[i] = (m0 + (row ^ (ks * 32))) * (K * 2) + kbase * 2 + ch * 16;
    }
#pragma unroll
    for (int i = 0; i < 4; ++i) {
      const int krow = 4 * (lw * 4 + i) + (lane >> 4), ch = (lane & 15) ^ hB(krow);
      voB[i] = (kbase + krow) * (N * 2) + n0 * 2 + ch * 16;
    }
    auto issue = [&](int kt, int slot) __attribute__((always_inline)) {
      const int soA = kt * (BK * 2), soB = kt * (BK * N * 2);
      LAS char* sb = lds + slot * SLOT;
#pragma unroll
      for (int i = 0; i < 4; ++i) {
        __builtin_amdgcn_raw_ptr_buffer_load_lds(rx, (LAS void*)(sb + (lw * 4 + i) * 1024), 16, voA[i], soA, 0, 0);
        __builtin_amdgcn_raw_ptr_buffer_load_lds(rw, (LAS void*)(sb + A_BYTES + (lw * 4 + i) * 1024), 16, voB[i], soB, 0, 0);
      }
    };
    unsigned pfd = 0;
    const int pg = idx >> 2, pj = (idx & 3) * 4 + lw;
    const int qvw = (kbase + pj * 4 + (lane >> 4)) * (N * 2) + ((lane & 15) >> 1) * 1024 + (((xcd >> 1) + ((lane & 15) >> 1)) & 3) * 256 + (lane & 1) * 128;
    const int qvx = (pj * 32 + (lane & 31)) * (K * 2) + kbase * 2;
    auto burst = [&](int kt) __attribute__((always_inline)) {
      asm volatile("buffer_load_dword %0, %1, %2, %3 offen" : "+v"(pfd) : "v"(qvw), "s"(rw), "s"(kt * (BK * N * 2)));
      asm volatile("buffer_load_dword %0, %1, %2, %3 offen" : "+v"(pfd) : "v"(qvx), "s"(rx), "s"(kt * (BK * 2)));
    };
    STAMP(0);
    const int rot = (tm + tn) & 3;
    issue(rot, 0); issue((rot + 1) & 3, 1);
    unsigned gdl = 0, gdh = 0;
    if (bid == LEADER) {
#pragma unroll
      for (int i = 1; i < NR; ++i) {
        const bool c = (lane & 7) == i;
        unsigned lo = (unsigned)(unsigned long long)p.ws[i], hi = (unsigned)((unsigned long long)p.ws[i] >> 32);
        asm volatile("" : "+s"(lo), "+s"(hi));
        gdl = c ? lo : gdl; gdh = c ? hi : gdh;
      }
      gdl = (lane & 7) == 0 ? (unsigned)(unsigned long long)p.ws[0] : gdl;
      gdh = (lane & 7) == 0 ? (unsigned)((unsigned long long)p.ws[0] >> 32) : gdh;
    }
    unsigned long long gd = (((unsigned long long)gdh << 32) | gdl) + (WS_AMAX + (size_t)((lane >> 3) * 8 + rank) * 128);
    asm volatile("" : "+v"(gd));
    wait_vm<PER>(); BAR;
    issue((rot + 2) & 3, 2); issue((rot + 3) & 3, 3);
    { const int kp = S + ((pg - S) & 7); if (kp < S + PF_LEAD) burst(kp); }
    wait_vm<2 * PER>(); BAR;
#pragma unroll
    for (int t = 1; t <= T - S; ++t) {
      const int kp = t + S - 1 + PF_LEAD;
      if (kp < T && (kp & 7) == pg) burst(kp);
      issue(t + S - 1, (t - 1) & (S - 1));
      wait_vm<2 * PER>(); BAR;
    }
    wait_vm<PER>(); BAR;
    asm volatile("s_waitcnt vmcnt(0)" : "+v"(pfd) :: "memory"); BAR;
    BAR;
    const unsigned call = __builtin_amdgcn_readfirstlane(s_call[0]);
    const unsigned par = call & 1u;
    const unsigned long long t0 = __builtin_amdgcn_s_memrealtime(); unsigned spins = 0;
    if (bid == LEADER) {
      auto rl = __builtin_amdgcn_make_buffer_rsrc((void*)(p.myws + WS_LOC), 0, NWG * 64, 0x00020000);
      unsigned v = 0; bool need = true, bad = false;
      for (;;) {
        if (need) {
          const u32x4 g0 = __builtin_amdgcn_raw_buffer_load_b128(rl, (lw * 64 + lane) * 64, 0, AUX_AGENT);
          const u32x4 g1 = __builtin_amdgcn_raw_buffer_load_b128(rl, (lw * 64 + lane) * 64 + 16, 0, AUX_AGENT);
          if (g0[1] == call && g0[3] == call && g1[1] == call && g1[3] == call) { v = max(max(g0[0], g0[2]), max(g1[0], g1[2])); need = false; }
        }
        if (!__any(need)) break;
        if (give_up(tmo, call, t0, spins)) { bad = true; break; }
      }
      if (!bad) {
        const unsigned wm = __float_as_uint(wave_max(__uint_as_float(v)));
        unsigned n = 0;
        if (lane == 0) {
          __hip_atomic_fetch_max(s_lead, wm, __ATOMIC_RELAXED, __HIP_MEMORY_SCOPE_WORKGROUP);
          n = __hip_atomic_fetch_add(s_lead + 1, 1u, __ATOMIC_RELAXED, __HIP_MEMORY_SCOPE_WORKGROUP);
        }
        if (__builtin_amdgcn_readfirstlane(n) == 3u) {
          const unsigned rm = *(volatile LAS unsigned*)s_lead;
          __hip_atomic_store((GAS unsigned long long*)(gd + par * 8192), ((unsigned long long)call << 32) | rm, __ATOMIC_RELAXED, __HIP_MEMORY_SCOPE_SYSTEM);
          STAMP(10);
        }
      }
    }
    if (lw == 0) {
      GAS unsigned long long* rg = (GAS unsigned long long*)(myws + WS_AMAX + par * 8192 + (xcd * 8 + (lane & 7)) * 128);
      unsigned v = 0; bool need = lane < NR, bad = false;
      for (;;) {
        if (need) {
          const unsigned long long g = __hip_atomic_load(rg, __ATOMIC_RELAXED, __HIP_MEMORY_SCOPE_SYSTEM);
          if ((unsigned)(g >> 32) == call) { v = (unsigned)g; need = false; }
        }
        if (!__any(need)) break;
        if (give_up(tmo, call, t0, spins)) { bad = true; break; }
      }
      const float amax = wave_max(__uint_as_float(v));
      float inv;
      if (amax > 1e-30f && amax < 1e30f) {
        float rc = __builtin_amdgcn_rcpf(amax);
        rc = fmaf(fmaf(-amax, rc, 1.0f), rc, rc);
        inv = 127.0f * rc;
      } else {
        const float scale = amax / 127.0f;
        inv = scale > 0.f ? 1.0f / scale : 0.f;
      }
      if (lane == 0) *s_g = (u32x2){__float_as_uint(inv), bad ? 2u : 1u};
      STAMP(5);
    } else if (lw >= 2) {
      auto ro = __builtin_amdgcn_make_buffer_rsrc((void*)p.myout, 0, N * NLOC, 0x00020000);
      while ((*s_g)[1] == 0u) __builtin_amdgcn_s_sleep(8);
      const int vo = bid * 8192 + (lw - 2) * 4096 + lane * 16;
      unsigned need = 0xfu;
      for (;;) {
        u32x4 v[4];
#pragma unroll
        for (int i = 0; i < 4; ++i) if (need & (1u << i)) v[i] = __builtin_amdgcn_raw_buffer_load_b128(ro, vo + i * 1024, 0, AUX_SYS);
#pragma unroll
        for (int i = 0; i < 4; ++i)
          if (need & (1u << i)) {
            unsigned z = 0;
#pragma unroll
            for (int j = 0; j < 4; ++j) { const unsigned t = v[i][j] ^ 0x80808080u; z |= (t - 0x01010101u) & ~t & 0x80808080u; }
            if (z == 0u) need &= ~(1u << i);
          }
        if (!__any(need != 0u)) break;
        if (give_up(tmo, call, t0, spins)) break;
      }
      if (lw == 2) STAMP(7);
    }
  } else {
    const int wr = wid >> 1, wc = wid & 1;
    const int g = lane >> 4, r = lane & 15, q = r >> 2, pp = lane & 3;
    {
      auto ro = __builtin_amdgcn_make_buffer_rsrc((void*)p.myout, 0, N * NLOC, 0x00020000);
      const u32x4 f = {0x80808080u, 0x80808080u, 0x80808080u, 0x80808080u};
      const int vo = bid * 8192 + wid * 2048 + lane * 16;
      __builtin_amdgcn_raw_buffer_store_b128(f, ro, vo, 0, AUX_SYS);
      __builtin_amdgcn_raw_buffer_store_b128(f, ro, vo + 1024, 0, AUX_SYS);
    }
    GAS unsigned* cnt = (GAS unsigned*)(myws + WS_CNT + bid * 64);
    const unsigned callv = __hip_atomic_load(cnt, __ATOMIC_RELAXED, __HIP_MEMORY_SCOPE_AGENT);
    if (threadIdx.x == 0) { *s_g = (u32x2){0u, 0u}; s_lead[0] = 0u; s_lead[1] = 0u; }
    unsigned base[6], cur[6];
#pragma unroll
    for (int k2 = 0; k2 < 2; ++k2) base[k2] = (wr * 64 + r) * 128 + (((4 * k2 + g) ^ ((r >> 1) & 7)) * 16);
    {
      const int krow = 8 * g + q, hx = hB(krow);
#pragma unroll
      for (int nb = 0; nb < 4; ++nb) base[2 + nb] = A_BYTES + krow * 256 + (((wc * 8 + 2 * nb + (pp >> 1)) ^ hx) * 16) + (pp & 1) * 8;
    }
    f32x4 acc[4][4];
#pragma unroll
    for (int i = 0; i < 4; ++i)
#pragma unroll
      for (int j = 0; j < 4; ++j) acc[i][j] = (f32x4){0.f, 0.f, 0.f, 0.f};
    s16x8 fa[2][2][4];
    s16x4 fb[2][2][4][2];
    auto rd1 = [&](int buf, int i) __attribute__((always_inline)) {
      const int k2 = i / 12, j = i % 12;
      if (j < 8) fb[buf][k2][j >> 1][j & 1] = __builtin_amdgcn_ds_read_tr16_b64_v4i16((LAS s16x4*)(lds + cur[2 + (j >> 1)] + (32 * k2 + 4 * (j & 1)) * 256));
      else fa[buf][k2][j - 8] = *(LAS s16x8*)(lds + cur[k2] + (j - 8) * 2048);
    };
    auto mf1 = [&](int buf, int i) __attribute__((always_inline)) {
      const int k2 = i >> 4, mb = (i >> 2) & 3, nb = i & 3;
      s16x8 b8 = __builtin_shufflevector(fb[buf][k2][nb][0], fb[buf][k2][nb][1], 0, 1, 2, 3, 4, 5, 6, 7);
      MF(acc[mb][nb], b8, fa[buf][k2][mb]);
    };
    int slot = 0;
    auto advance = [&]() __attribute__((always_inline)) {
      slot = (slot + 1) & (S - 1);
      const unsigned so = slot * SLOT;
#pragma unroll
      for (int i = 0; i < 6; ++i) cur[i] = base[i] + so;
    };
    auto stage = [&](int buf) __attribute__((always_inline)) {
      BAR; SB0;
#pragma unroll
      for (int i = 0; i < 32; ++i) {
        mf1(buf ^ 1, i);
        if (i < 24) rd1(buf, i);
        if (i == 28) advance();
        if (i == 30) wait_lgkm0();
        SB0;
      }
    };
#pragma unroll
    for (int i = 0; i < 6; ++i) cur[i] = base[i];
    BAR; SB0;
    if (wid == 0) STAMP(1);
#pragma unroll
    for (int i = 0; i < 24; ++i) rd1(0, i);
    advance();
    wait_lgkm0(); SB0;
    for (int t = 1; t < T - 5; t += 2) { stage(1); stage(0); }
    stage(1);
    wait_vm<0>();
    const unsigned call = __builtin_amdgcn_readfirstlane(callv) + 1u, par = call & 1u;
    if (threadIdx.x == 0) s_call[0] = call;
    auto rxch = __builtin_amdgcn_make_buffer_rsrc((void*)(p.myws + WS_XCH), 0, NWG * XCH_WG, 0x00020000);
    const int vos = (bid ^ 1) * XCH_WG + wid * 8192 + lane * 16;
    float lm = 0.f, lr = 0.f;
    auto send = [&](int c) __attribute__((always_inline)) {
      if (c < 4) {
#pragma unroll
        for (int e = 2 * c; e < 2 * c + 2; ++e) {
          const f32x4 a = acc[2 + (e >> 2)][e & 3];
          lm = fmaxf(fmaxf(lm, fmaxf(fabsf(a[0]), fabsf(a[1]))), fmaxf(fabsf(a[2]), fabsf(a[3])));
        }
        if (c == 3) {
          lr = lm > 0.f ? __builtin_amdgcn_rcpf(lm) : 0.f;
          __builtin_amdgcn_raw_buffer_store_b32((__float_as_uint(lm) & ~1u) | par, rxch, (bid ^ 1) * XCH_WG + wid * 8192 + 4096 + lane * 4, 0, AUX_AGENT);
        }
      } else {
        const int e = 2 * (c - 4);
        const f32x4 a = acc[2 + (e >> 2)][e & 3] * lr, b = acc[2 + (e >> 2)][(e & 3) + 1] * lr;
        const u32x4 v = {enc16(a[0], a[1], par), enc16(a[2], a[3], par), enc16(b[0], b[1], par), enc16(b[2], b[3], par)};
        __builtin_amdgcn_raw_buffer_store_b128(v, rxch, vos + (c - 4) * 1024, 0, AUX_AGENT);
      }
    };
    s16x8 ta[2][2][2];
    s16x4 tb[2][2][4][2];
    auto trd = [&](int u, int j) __attribute__((always_inline)) {
      const int mb0 = u < 4 ? 2 : 0, so = (u & 3) * SLOT, k2 = j / 10, e = j % 10;
      if (e < 8) tb[u & 1][k2][e >> 1][e & 1] = __builtin_amdgcn_ds_read_tr16_b64_v4i16((LAS s16x4*)(lds + (base[2 + (e >> 1)] + so) + (32 * k2 + 4 * (e & 1)) * 256));
      else ta[u & 1][k2][e - 8] = *(LAS s16x8*)(lds + (base[k2] + so) + (mb0 + e - 8) * 2048);
    };
    auto tmf = [&](int u, int i) __attribute__((always_inline)) {
      const int mb0 = u < 4 ? 2 : 0, k2 = i >> 3, mb = (i >> 2) & 1, nb = i & 3;
      s16x8 b8 = __builtin_shufflevector(tb[u & 1][k2][nb][0], tb[u & 1][k2][nb][1], 0, 1, 2, 3, 4, 5, 6, 7);
      MF(acc[mb0 + mb][nb], b8, ta[u & 1][k2][mb]);
    };
    const int vox = bid * XCH_WG + wid * 8192 + lane * 16, voxs = bid * XCH_WG + wid * 8192 + 4096 + lane * 4;
    static_assert((T - 4) % S == 0, "tail slot mapping");
    BAR; SB0;
#pragma unroll
    for (int i = 0; i < 32; ++i) {
      mf1(1, i);
      if (i < 20) trd(0, i);
      if (i == 30) wait_lgkm0();
      SB0;
    }
#pragma unroll
    for (int u = 0; u < 8; ++u) {
      if (u < 3) { BAR; SB0; }
#pragma unroll
      for (int i = 0; i < 16; ++i) {
        tmf(u, i);
        if (u < 7 && i < 10) { trd(u + 1, 2 * i); trd(u + 1, 2 * i + 1); }
        SB0;
        if (u == 4 && i >= 6 && i < 14) { send(i - 6); SB0; }
        if (i == 14) { wait_lgkm0(); SB0; }
      }
    }
    asm volatile("s_nop 15\n\ts_nop 7" ::: "memory");
    SB0;
    if (wid == 0) STAMP(2);
    bool bad = false;
    {
      const unsigned pm = par ? 0xffffffffu : 0u;
      const unsigned long long t0 = __builtin_amdgcn_s_memrealtime(); unsigned spins = 0;
      u32x4 xv[4]; unsigned xs;
      for (;;) {
#pragma unroll
        for (int c = 0; c < 4; ++c) xv[c] = __builtin_amdgcn_raw_buffer_load_b128(rxch, vox + c * 1024, 0, AUX_AGENT);
        xs = __builtin_amdgcn_raw_buffer_load_b32(rxch, voxs, 0, AUX_AGENT);
        const unsigned o = (xs ^ pm) | tagbits(xv[0], pm) | tagbits(xv[1], pm) | tagbits(xv[2], pm) | tagbits(xv[3], pm);
        if (__all((o & 1u) == 0u)) break;
        if (give_up(tmo, call, t0, spins)) { bad = true; break; }
      }
      const float f = __uint_as_float(xs & ~1u) * (1.0f / 32767.0f);
#pragma unroll
      for (int c = 0; c < 8; ++c) {
        const unsigned e0 = xv[c >> 1][(c & 1) * 2], e1 = xv[c >> 1][(c & 1) * 2 + 1];
        f32x4& a = acc[c >> 2][c & 3];
        a[0] = fmaf(lo16(e0), f, a[0]); a[1] = fmaf(hi16(e0), f, a[1]); a[2] = fmaf(lo16(e1), f, a[2]); a[3] = fmaf(hi16(e1), f, a[3]);
      }
    }
    float m = 0.f;
#pragma unroll
    for (int i = 0; i < 2; ++i)
#pragma unroll
      for (int nb = 0; nb < 4; ++nb)
#pragma unroll
        for (int j = 0; j < 4; ++j) m = fmaxf(m, fabsf(acc[i][nb][j]));
    m = wave_max(m);
    if (lane == 0) __hip_atomic_store((GAS unsigned long long*)(myws + WS_LOC + bid * 64 + wid * 8), ((unsigned long long)call << 32) | __float_as_uint(m),
                                      __ATOMIC_RELAXED, __HIP_MEMORY_SCOPE_AGENT);
    if (wid == 0) { STAMP(3); STAMP(4); }
    BAR;
    const unsigned dest = ((unsigned)n0 / NLOC) & (NR - 1);
    if (threadIdx.x == 0) __hip_atomic_store(cnt, call, __ATOMIC_RELAXED, __HIP_MEMORY_SCOPE_AGENT);
    auto rd = __builtin_amdgcn_make_buffer_rsrc((void*)p.out[dest], 0, N * NLOC, 0x00020000);
    const int vod = (rank * MLOC + m0 + wr * 64 + ks * 32 + r) * NLOC + (n0 & (NLOC - 1)) + wc * 64 + g * 16;
    const float inv = wait_amax(bad);
#pragma unroll
    for (int i = 0; i < 2; ++i) {
      unsigned d[4];
#pragma unroll
      for (int nb = 0; nb < 4; ++nb) d[nb] = q4(acc[i][nb][0], acc[i][nb][1], acc[i][nb][2], acc[i][nb][3], inv);
      u32x4 o = xpose(d[0], d[1], d[2], d[3]);
      if (bad) o = (u32x4){0x80808080u, 0x80808080u, 0x80808080u, 0x80808080u};
      __builtin_amdgcn_raw_buffer_store_b128(o, rd, vod + i * 16 * NLOC, 0, AUX_SYS);
    }
    if (wid == 0) STAMP(6);
  }
#endif
}

}

extern "C" void kernel_launch(int rank, int nranks, ncclComm_t comm,
                              hipStream_t stream,
                              void* const* d_in, const int* in_sizes, int n_in,
                              void* d_out, int out_size,
                              const int* in_dtypes, int out_dtype,
                              void* const* d_out_all,
                              void* const* d_ws_all, size_t ws_bytes) {
  (void)comm; (void)in_dtypes; (void)out_dtype; (void)d_out;
  if (nranks != NR || n_in != 2 || in_sizes[0] != MLOC * K || in_sizes[1] != K * N || out_size != N * NLOC ||
      ws_bytes < WS_NEED || (unsigned)rank >= (unsigned)NR) {
    fprintf(stderr, "kernel_launch: unexpected problem shape\n");
    return;
  }
  static int ready = 0;
  if (!ready) {
    HIP_CHECK(hipFuncSetAttribute((const void*)gemm_a2a_q8, hipFuncAttributeMaxDynamicSharedMemorySize, LDS_BYTES));
    int dev = 0, cus = 0, per_cu = 0;
    HIP_CHECK(hipGetDevice(&dev));
    HIP_CHECK(hipDeviceGetAttribute(&cus, hipDeviceAttributeMultiprocessorCount, dev));
    HIP_CHECK(hipOccupancyMaxActiveBlocksPerMultiprocessor(&per_cu, (const void*)gemm_a2a_q8, 512, LDS_BYTES));
    if (cus * per_cu < NWG) {
      fprintf(stderr, "kernel_launch: %d workgroups cannot be co-resident (%d CUs x %d)\n", NWG, cus, per_cu);
      return;
    }
    ready = 1;
  }
  Params p;
  p.x = d_in[0]; p.w = d_in[1]; p.rank = rank; p.pad = 0;
  p.myout = (char*)d_out_all[rank]; p.myws = (char*)d_ws_all[rank];
  for (int i = 0; i < NR; ++i) { p.out[i] = (char*)d_out_all[i]; p.ws[i] = (char*)d_ws_all[i]; }
  hipLaunchKernelGGL(gemm_a2a_q8, dim3(NWG), dim3(512), LDS_BYTES, stream, p);
  HIP_CHECK(hipGetLastError());
}
```

```cpp
#include <hip/hip_runtime.h>
#include <rccl/rccl.h>
#include <cstdio>

#define HIP_CHECK(expr) do { hipError_t _e = (expr); if (_e != hipSuccess) { \
    fprintf(stderr, "HIP error %s at %s:%d\n", hipGetErrorString(_e), __FILE__, __LINE__); \
    return; } } while (0)

namespace {

typedef short s16x4 __attribute__((ext_vector_type(4)));
typedef short s16x8 __attribute__((ext_vector_type(8)));
typedef float f32x4 __attribute__((ext_vector_type(4)));
typedef unsigned u32x2 __attribute__((ext_vector_type(2)));
typedef unsigned u32x4 __attribute__((ext_vector_type(4)));
#define LAS __attribute__((address_space(3)))
#define GAS __attribute__((address_space(1)))

constexpr int NR = 8;
constexpr int MLOC = 512, N = 4096, K = 4096, NLOC = 512;
constexpr int BM = 128, BN = 128, BK = 64, KS = 2;
constexpr int T = K / KS / BK;
constexpr int S = 4;
constexpr int A_BYTES = BM * BK * 2, B_BYTES = BK * BN * 2, SLOT = A_BYTES + B_BYTES;
constexpr int LDS_MISC = S * SLOT;
constexpr int LDS_BYTES = S * SLOT + 256;
constexpr int NWG = 256;
constexpr int PER = 8;

constexpr size_t WS_TMO = 0;
constexpr size_t WS_CNT = 4096;
constexpr size_t WS_LOC = 65536;
constexpr size_t WS_AMAX = 131072;
constexpr size_t WS_DBG = 262144;
constexpr size_t WS_XCH = 1u << 20;
constexpr int XCH_WG = 32768;
constexpr size_t WS_NEED = WS_XCH + (size_t)NWG * XCH_WG;
constexpr unsigned long long DEADLINE = 50000000ull;

constexpr int LEADER = 0;

struct Params {
  const void* x; const void* w;
  char* myout; char* myws;
  int rank, pad;
  char* out[NR]; char* ws[NR];
};

#define MEMFENCE asm volatile("" ::: "memory")
#define SB0 __builtin_amdgcn_sched_barrier(0)
#define BAR do { MEMFENCE; __builtin_amdgcn_s_barrier(); MEMFENCE; } while (0)
#define MF(acc, a, b) asm volatile("v_mfma_f32_16x16x32_bf16 %0, %1, %2, %0" : "+v"(acc) : "v"(a), "v"(b))
template <int N_> __device__ __forceinline__ void wait_vm() { asm volatile("s_waitcnt vmcnt(%0)" :: "i"(N_) : "memory"); }
__device__ __forceinline__ void wait_lgkm0() { MEMFENCE; __builtin_amdgcn_s_waitcnt(0xC07F); MEMFENCE; }
__device__ __forceinline__ int hB(int row) { return ((row & 3) | (((row >> 3) & 1) << 2)) << 1; }
constexpr int AUX_AGENT = 16, AUX_SYS = 17;
constexpr float MAGIC = 12582912.0f;

#define STAMP(i) do { } while (0)

__device__ __forceinline__ float wave_max(float f) {
  unsigned v = __float_as_uint(f);
  v = max(v, (unsigned)__builtin_amdgcn_update_dpp(0, (int)v, 0xB1, 0xf, 0xf, false));
  v = max(v, (unsigned)__builtin_amdgcn_update_dpp(0, (int)v, 0x4E, 0xf, 0xf, false));
  v = max(v, (unsigned)__builtin_amdgcn_update_dpp(0, (int)v, 0x141, 0xf, 0xf, false));
  v = max(v, (unsigned)__builtin_amdgcn_update_dpp(0, (int)v, 0x140, 0xf, 0xf, false));
  const unsigned a = __builtin_amdgcn_readlane(v, 0), b = __builtin_amdgcn_readlane(v, 16);
  const unsigned c = __builtin_amdgcn_readlane(v, 32), d = __builtin_amdgcn_readlane(v, 48);
  return __uint_as_float(max(max(a, b), max(c, d)));
}

__device__ __forceinline__ bool give_up(GAS unsigned* tmo, unsigned call, unsigned long long t0, unsigned& spins) {
  __builtin_amdgcn_s_sleep(2);
  if ((++spins & 15u) != 0u) return false;
  if ((unsigned)__builtin_amdgcn_readfirstlane(__hip_atomic_load(tmo, __ATOMIC_RELAXED, __HIP_MEMORY_SCOPE_AGENT)) == call) return true;
  if (__builtin_amdgcn_s_memrealtime() - t0 > DEADLINE) {
    __hip_atomic_store(tmo, call, __ATOMIC_RELAXED, __HIP_MEMORY_SCOPE_AGENT);
    return true;
  }
  return false;
}
__device__ __forceinline__ unsigned enc16(float a, float b, unsigned tag) {
  return (__builtin_bit_cast(unsigned, __builtin_amdgcn_cvt_pknorm_i16(a, b)) & ~1u) | tag;
}
__device__ __forceinline__ float lo16(unsigned v) { return (float)(short)(v & 0xfffeu); }
__device__ __forceinline__ float hi16(unsigned v) { return (float)((int)v >> 16); }
__device__ __forceinline__ unsigned tagbits(const u32x4& v, unsigned pm) { return (v[0] ^ pm) | (v[1] ^ pm) | (v[2] ^ pm) | (v[3] ^ pm); }
__device__ __forceinline__ unsigned q4(float a, float b, float c, float d, float f) {
  const unsigned b0 = __float_as_uint(__builtin_amdgcn_fmed3f(fmaf(a, f, MAGIC), MAGIC - 127.0f, MAGIC + 127.0f));
  const unsigned b1 = __float_as_uint(__builtin_amdgcn_fmed3f(fmaf(b, f, MAGIC), MAGIC - 127.0f, MAGIC + 127.0f));
  const unsigned b2 = __float_as_uint(__builtin_amdgcn_fmed3f(fmaf(c, f, MAGIC), MAGIC - 127.0f, MAGIC + 127.0f));
  const unsigned b3 = __float_as_uint(__builtin_amdgcn_fmed3f(fmaf(d, f, MAGIC), MAGIC - 127.0f, MAGIC + 127.0f));
  return __builtin_amdgcn_perm(__builtin_amdgcn_perm(b3, b2, 0x0c0c0400u), __builtin_amdgcn_perm(b1, b0, 0x0c0c0400u), 0x05040100u);
}
__device__ __forceinline__ u32x4 xpose(unsigned d0, unsigned d1, unsigned d2, unsigned d3) {
  { auto t = __builtin_amdgcn_permlane32_swap(d0, d2, false, false); d0 = t[0]; d2 = t[1]; }
  { auto t = __builtin_amdgcn_permlane32_swap(d1, d3, false, false); d1 = t[0]; d3 = t[1]; }
  { auto t = __builtin_amdgcn_permlane16_swap(d0, d1, false, false); d0 = t[0]; d1 = t[1]; }
  { auto t = __builtin_amdgcn_permlane16_swap(d2, d3, false, false); d2 = t[0]; d3 = t[1]; }
  return (u32x4){d0, d1, d2, d3};
}

__global__ __launch_bounds__(512, 2) void gemm_a2a_q8(const Params p) {
#if defined(__HIP_DEVICE_COMPILE__)
  extern __shared__ __attribute__((aligned(16))) char smem[];
  LAS char* lds = (LAS char*)smem;
  LAS unsigned* s_call = (LAS unsigned*)(lds + LDS_MISC);
  volatile LAS u32x2* s_g = (volatile LAS u32x2*)(lds + LDS_MISC + 16);
  LAS unsigned* s_lead = (LAS unsigned*)(lds + LDS_MISC + 32);
  const int bid = blockIdx.x;
  const int xcd = bid & 7, idx = bid >> 3;
  const int ks = xcd & 1, tm = idx & 3, tn = idx >> 2;
  const int m0 = tm * BM, n0 = tn * NLOC + (((xcd >> 1) + tn) & 3) * BN;
  const int kbase = ks * (K / KS);
  const int wid = __builtin_amdgcn_readfirstlane(threadIdx.x >> 6), lane = threadIdx.x & 63;
  const int rank = p.rank;
  GAS char* myws = (GAS char*)p.myws;
  GAS unsigned* tmo = (GAS unsigned*)(myws + WS_TMO);
  auto wait_amax = [&](bool& bad) __attribute__((always_inline)) -> float {
    u32x2 g;
    do { g = *s_g; } while (g[1] == 0u);
    bad = bad || g[1] != 1u;
    return __uint_as_float(g[0]);
  };

  if (wid >= 4) {
    const int lw = wid - 4;
    auto rx = __builtin_amdgcn_make_buffer_rsrc((void*)p.x, 0, MLOC * K * 2, 0x00020000);
    auto rw = __builtin_amdgcn_make_buffer_rsrc((void*)p.w, 0, K * N * 2, 0x00020000);
    int voA[4], voB[4];
#pragma unroll
    for (int i = 0; i < 4; ++i) {
      const int row = 8 * (lw * 4 + i) + (lane >> 3), ch = (lane & 7) ^ ((row >> 1) & 7);
      voA[i] = (m0 + (row ^ (ks * 32))) * (K * 2) + kbase * 2 + ch * 16;
    }
#pragma unroll
    for (int i = 0; i < 4; ++i) {
      const int krow = 4 * (lw * 4 + i) + (lane >> 4), ch = (lane & 15) ^ hB(krow);
      voB[i] = (kbase + krow) * (N * 2) + n0 * 2 + ch * 16;
    }
    auto issue = [&](int kt, int slot) __attribute__((always_inline)) {
      const int soA = kt * (BK * 2), soB = kt * (BK * N * 2);
      LAS char* sb = lds + slot * SLOT;
#pragma unroll
      for (int i = 0; i < 4; ++i) {
        __builtin_amdgcn_raw_ptr_buffer_load_lds(rx, (LAS void*)(sb + (lw * 4 + i) * 1024), 16, voA[i], soA, 0, 0);
        __builtin_amdgcn_raw_ptr_buffer_load_lds(rw, (LAS void*)(sb + A_BYTES + (lw * 4 + i) * 1024), 16, voB[i], soB, 0, 0);
      }
    };
    const int rot = tm + tn;
    auto kt_of = [&](int q) __attribute__((always_inline)) -> int { return (q & ~3) + ((rot + (q >> 3) + q) & 3); };
    STAMP(0);
    issue(kt_of(0), 0); issue(kt_of(1), 1);
    unsigned gdl = 0, gdh = 0;
    if (bid == LEADER) {
#pragma unroll
      for (int i = 1; i < NR; ++i) {
        const bool c = (lane & 7) == i;
        unsigned lo = (unsigned)(unsigned long long)p.ws[i], hi = (unsigned)((unsigned long long)p.ws[i] >> 32);
        asm volatile("" : "+s"(lo), "+s"(hi));
        gdl = c ? lo : gdl; gdh = c ? hi : gdh;
      }
      gdl = (lane & 7) == 0 ? (unsigned)(unsigned long long)p.ws[0] : gdl;
      gdh = (lane & 7) == 0 ? (unsigned)((unsigned long long)p.ws[0] >> 32) : gdh;
    }
    unsigned long long gd = (((unsigned long long)gdh << 32) | gdl) + (WS_AMAX + (size_t)((lane >> 3) * 8 + rank) * 128);
    asm volatile("" : "+v"(gd));
    wait_vm<PER>(); BAR;
    issue(kt_of(2), 2); issue(kt_of(3), 3);
    wait_vm<2 * PER>(); BAR;
#pragma unroll
    for (int t = 1; t <= T - S; ++t) {
      issue(kt_of(t + S - 1), (t - 1) & (S - 1));
      wait_vm<2 * PER>(); BAR;
    }
    wait_vm<PER>(); BAR;
    wait_vm<0>(); BAR;
    BAR;
    const unsigned call = __builtin_amdgcn_readfirstlane(s_call[0]);
    const unsigned par = call & 1u;
    const unsigned long long t0 = __builtin_amdgcn_s_memrealtime(); unsigned spins = 0;
    if (bid == LEADER) {
      auto rl = __builtin_amdgcn_make_buffer_rsrc((void*)(p.myws + WS_LOC), 0, NWG * 64, 0x00020000);
      unsigned v = 0; bool need = true, bad = false;
      for (;;) {
        if (need) {
          const u32x4 g0 = __builtin_amdgcn_raw_buffer_load_b128(rl, (lw * 64 + lane) * 64, 0, AUX_AGENT);
          const u32x4 g1 = __builtin_amdgcn_raw_buffer_load_b128(rl, (lw * 64 + lane) * 64 + 16, 0, AUX_AGENT);
          if (g0[1] == call && g0[3] == call && g1[1] == call && g1[3] == call) { v = max(max(g0[0], g0[2]), max(g1[0], g1[2])); need = false; }
        }
        if (!__any(need)) break;
        if (give_up(tmo, call, t0, spins)) { bad = true; break; }
      }
      if (!bad) {
        const unsigned wm = __float_as_uint(wave_max(__uint_as_float(v)));
        unsigned n = 0;
        if (lane == 0) {
          __hip_atomic_fetch_max(s_lead, wm, __ATOMIC_RELAXED, __HIP_MEMORY_SCOPE_WORKGROUP);
          n = __hip_atomic_fetch_add(s_lead + 1, 1u, __ATOMIC_RELAXED, __HIP_MEMORY_SCOPE_WORKGROUP);
        }
        if (__builtin_amdgcn_readfirstlane(n) == 3u) {
          const unsigned rm = *(volatile LAS unsigned*)s_lead;
          __hip_atomic_store((GAS unsigned long long*)(gd + par * 8192), ((unsigned long long)call << 32) | rm, __ATOMIC_RELAXED, __HIP_MEMORY_SCOPE_SYSTEM);
          STAMP(10);
        }
      }
    }
    if (lw == 0) {
      GAS unsigned long long* rg = (GAS unsigned long long*)(myws + WS_AMAX + par * 8192 + (xcd * 8 + (lane & 7)) * 128);
      unsigned v = 0; bool need = lane < NR, bad = false;
      for (;;) {
        if (need) {
          const unsigned long long g = __hip_atomic_load(rg, __ATOMIC_RELAXED, __HIP_MEMORY_SCOPE_SYSTEM);
          if ((unsigned)(g >> 32) == call) { v = (unsigned)g; need = false; }
        }
        if (!__any(need)) break;
        if (give_up(tmo, call, t0, spins)) { bad = true; break; }
      }
      const float amax = wave_max(__uint_as_float(v));
      float inv;
      if (amax > 1e-30f && amax < 1e30f) {
        float rc = __builtin_amdgcn_rcpf(amax);
        rc = fmaf(fmaf(-amax, rc, 1.0f), rc, rc);
        inv = 127.0f * rc;
      } else {
        const float scale = amax / 127.0f;
        inv = scale > 0.f ? 1.0f / scale : 0.f;
      }
      if (lane == 0) *s_g = (u32x2){__float_as_uint(inv), bad ? 2u : 1u};
      STAMP(5);
    } else if (lw >= 2) {
      auto ro = __builtin_amdgcn_make_buffer_rsrc((void*)p.myout, 0, N * NLOC, 0x00020000);
      while ((*s_g)[1] == 0u) __builtin_amdgcn_s_sleep(8);
      const int vo = bid * 8192 + (lw - 2) * 4096 + lane * 16;
      unsigned need = 0xfu;
      for (;;) {
        u32x4 v[4];
#pragma unroll
        for (int i = 0; i < 4; ++i) if (need & (1u << i)) v[i] = __builtin_amdgcn_raw_buffer_load_b128(ro, vo + i * 1024, 0, AUX_SYS);
#pragma unroll
        for (int i = 0; i < 4; ++i)
          if (need & (1u << i)) {
            unsigned z = 0;
#pragma unroll
            for (int j = 0; j < 4; ++j) { const unsigned t = v[i][j] ^ 0x80808080u; z |= (t - 0x01010101u) & ~t & 0x80808080u; }
            if (z == 0u) need &= ~(1u << i);
          }
        if (!__any(need != 0u)) break;
        if (give_up(tmo, call, t0, spins)) break;
      }
      if (lw == 2) STAMP(7);
    }
  } else {
    const int wr = wid >> 1, wc = wid & 1;
    const int g = lane >> 4, r = lane & 15, q = r >> 2, pp = lane & 3;
    {
      auto ro = __builtin_amdgcn_make_buffer_rsrc((void*)p.myout, 0, N * NLOC, 0x00020000);
      const u32x4 f = {0x80808080u, 0x80808080u, 0x80808080u, 0x80808080u};
      const int vo = bid * 8192 + wid * 2048 + lane * 16;
      __builtin_amdgcn_raw_buffer_store_b128(f, ro, vo, 0, AUX_SYS);
      __builtin_amdgcn_raw_buffer_store_b128(f, ro, vo + 1024, 0, AUX_SYS);
    }
    GAS unsigned* cnt = (GAS unsigned*)(myws + WS_CNT + bid * 64);
    const unsigned callv = __hip_atomic_load(cnt, __ATOMIC_RELAXED, __HIP_MEMORY_SCOPE_AGENT);
    if (threadIdx.x == 0) { *s_g = (u32x2){0u, 0u}; s_lead[0] = 0u; s_lead[1] = 0u; }
    unsigned base[6], cur[6];
#pragma unroll
    for (int k2 = 0; k2 < 2; ++k2) base[k2] = (wr * 64 + r) * 128 + (((4 * k2 + g) ^ ((r >> 1) & 7)) * 16);
    {
      const int krow = 8 * g + q, hx = hB(krow);
#pragma unroll
      for (int nb = 0; nb < 4; ++nb) base[2 + nb] = A_BYTES + krow * 256 + (((wc * 8 + 2 * nb + (pp >> 1)) ^ hx) * 16) + (pp & 1) * 8;
    }
    f32x4 acc[4][4];
#pragma unroll
    for (int i = 0; i < 4; ++i)
#pragma unroll
      for (int j = 0; j < 4; ++j) acc[i][j] = (f32x4){0.f, 0.f, 0.f, 0.f};
    s16x8 fa[2][2][4];
    s16x4 fb[2][2][4][2];
    auto rd1 = [&](int buf, int i) __attribute__((always_inline)) {
      const int k2 = i / 12, j = i % 12;
      if (j < 8) fb[buf][k2][j >> 1][j & 1] = __builtin_amdgcn_ds_read_tr16_b64_v4i16((LAS s16x4*)(lds + cur[2 + (j >> 1)] + (32 * k2 + 4 * (j & 1)) * 256));
      else fa[buf][k2][j - 8] = *(LAS s16x8*)(lds + cur[k2] + (j - 8) * 2048);
    };
    auto mf1 = [&](int buf, int i) __attribute__((always_inline)) {
      const int k2 = i >> 4, mb = (i >> 2) & 3, nb = i & 3;
      s16x8 b8 = __builtin_shufflevector(fb[buf][k2][nb][0], fb[buf][k2][nb][1], 0, 1, 2, 3, 4, 5, 6, 7);
      MF(acc[mb][nb], b8, fa[buf][k2][mb]);
    };
    int slot = 0;
    auto advance = [&]() __attribute__((always_inline)) {
      slot = (slot + 1) & (S - 1);
      const unsigned so = slot * SLOT;
#pragma unroll
      for (int i = 0; i < 6; ++i) cur[i] = base[i] + so;
    };
    auto stage = [&](int buf) __attribute__((always_inline)) {
      BAR; SB0;
#pragma unroll
      for (int i = 0; i < 32; ++i) {
        mf1(buf ^ 1, i);
        if (i < 24) rd1(buf, i);
        if (i == 28) advance();
        if (i == 30) wait_lgkm0();
        SB0;
      }
    };
#pragma unroll
    for (int i = 0; i < 6; ++i) cur[i] = base[i];
    BAR; SB0;
    if (wid == 0) STAMP(1);
#pragma unroll
    for (int i = 0; i < 24; ++i) rd1(0, i);
    advance();
    wait_lgkm0(); SB0;
    for (int t = 1; t < T - 5; t += 2) { stage(1); stage(0); }
    stage(1);
    wait_vm<0>();
    const unsigned call = __builtin_amdgcn_readfirstlane(callv) + 1u, par = call & 1u;
    if (threadIdx.x == 0) s_call[0] = call;
    auto rxch = __builtin_amdgcn_make_buffer_rsrc((void*)(p.myws + WS_XCH), 0, NWG * XCH_WG, 0x00020000);
    const int vos = (bid ^ 1) * XCH_WG + wid * 8192 + lane * 16;
    float lm = 0.f, lr = 0.f;
    auto send = [&](int c) __attribute__((always_inline)) {
      if (c < 4) {
#pragma unroll
        for (int e = 2 * c; e < 2 * c + 2; ++e) {
          const f32x4 a = acc[2 + (e >> 2)][e & 3];
          lm = fmaxf(fmaxf(lm, fmaxf(fabsf(a[0]), fabsf(a[1]))), fmaxf(fabsf(a[2]), fabsf(a[3])));
        }
        if (c == 3) {
          lr = lm > 0.f ? __builtin_amdgcn_rcpf(lm) : 0.f;
          __builtin_amdgcn_raw_buffer_store_b32((__float_as_uint(lm) & ~1u) | par, rxch, (bid ^ 1) * XCH_WG + wid * 8192 + 4096 + lane * 4, 0, AUX_AGENT);
        }
      } else {
        const int e = 2 * (c - 4);
        const f32x4 a = acc[2 + (e >> 2)][e & 3] * lr, b = acc[2 + (e >> 2)][(e & 3) + 1] * lr;
        const u32x4 v = {enc16(a[0], a[1], par), enc16(a[2], a[3], par), enc16(b[0], b[1], par), enc16(b[2], b[3], par)};
        __builtin_amdgcn_raw_buffer_store_b128(v, rxch, vos + (c - 4) * 1024, 0, AUX_AGENT);
      }
    };
    s16x8 ta[2][2][2];
    s16x4 tb[2][2][4][2];
    auto trd = [&](int u, int j) __attribute__((always_inline)) {
      const int mb0 = u < 4 ? 2 : 0, so = (u & 3) * SLOT, k2 = j / 10, e = j % 10;
      if (e < 8) tb[u & 1][k2][e >> 1][e & 1] = __builtin_amdgcn_ds_read_tr16_b64_v4i16((LAS s16x4*)(lds + (base[2 + (e >> 1)] + so) + (32 * k2 + 4 * (e & 1)) * 256));
      else ta[u & 1][k2][e - 8] = *(LAS s16x8*)(lds + (base[k2] + so) + (mb0 + e - 8) * 2048);
    };
    auto tmf = [&](int u, int i) __attribute__((always_inline)) {
      const int mb0 = u < 4 ? 2 : 0, k2 = i >> 3, mb = (i >> 2) & 1, nb = i & 3;
      s16x8 b8 = __builtin_shufflevector(tb[u & 1][k2][nb][0], tb[u & 1][k2][nb][1], 0, 1, 2, 3, 4, 5, 6, 7);
      MF(acc[mb0 + mb][nb], b8, ta[u & 1][k2][mb]);
    };
    const int vox = bid * XCH_WG + wid * 8192 + lane * 16, voxs = bid * XCH_WG + wid * 8192 + 4096 + lane * 4;
    static_assert((T - 4) % S == 0, "tail slot mapping");
    BAR; SB0;
#pragma unroll
    for (int i = 0; i < 32; ++i) {
      mf1(1, i);
      if (i < 20) trd(0, i);
      if (i == 30) wait_lgkm0();
      SB0;
    }
#pragma unroll
    for (int u = 0; u < 8; ++u) {
      if (u < 3) { BAR; SB0; }
#pragma unroll
      for (int i = 0; i < 16; ++i) {
        tmf(u, i);
        if (u < 7 && i < 10) { trd(u + 1, 2 * i); trd(u + 1, 2 * i + 1); }
        SB0;
        if (u == 4 && i >= 6 && i < 14) { send(i - 6); SB0; }
        if (i == 14) { wait_lgkm0(); SB0; }
      }
    }
    asm volatile("s_nop 15\n\ts_nop 7" ::: "memory");
    SB0;
    if (wid == 0) STAMP(2);
    bool bad = false;
    {
      const unsigned pm = par ? 0xffffffffu : 0u;
      const unsigned long long t0 = __builtin_amdgcn_s_memrealtime(); unsigned spins = 0;
      u32x4 xv[4]; unsigned xs;
      for (;;) {
#pragma unroll
        for (int c = 0; c < 4; ++c) xv[c] = __builtin_amdgcn_raw_buffer_load_b128(rxch, vox + c * 1024, 0, AUX_AGENT);
        xs = __builtin_amdgcn_raw_buffer_load_b32(rxch, voxs, 0, AUX_AGENT);
        const unsigned o = (xs ^ pm) | tagbits(xv[0], pm) | tagbits(xv[1], pm) | tagbits(xv[2], pm) | tagbits(xv[3], pm);
        if (__all((o & 1u) == 0u)) break;
        if (give_up(tmo, call, t0, spins)) { bad = true; break; }
      }
      const float f = __uint_as_float(xs & ~1u) * (1.0f / 32767.0f);
#pragma unroll
      for (int c = 0; c < 8; ++c) {
        const unsigned e0 = xv[c >> 1][(c & 1) * 2], e1 = xv[c >> 1][(c & 1) * 2 + 1];
        f32x4& a = acc[c >> 2][c & 3];
        a[0] = fmaf(lo16(e0), f, a[0]); a[1] = fmaf(hi16(e0), f, a[1]); a[2] = fmaf(lo16(e1), f, a[2]); a[3] = fmaf(hi16(e1), f, a[3]);
      }
    }
    float m = 0.f;
#pragma unroll
    for (int i = 0; i < 2; ++i)
#pragma unroll
      for (int nb = 0; nb < 4; ++nb)
#pragma unroll
        for (int j = 0; j < 4; ++j) m = fmaxf(m, fabsf(acc[i][nb][j]));
    m = wave_max(m);
    if (lane == 0) __hip_atomic_store((GAS unsigned long long*)(myws + WS_LOC + bid * 64 + wid * 8), ((unsigned long long)call << 32) | __float_as_uint(m),
                                      __ATOMIC_RELAXED, __HIP_MEMORY_SCOPE_AGENT);
    if (wid == 0) { STAMP(3); STAMP(4); }
    BAR;
    const unsigned dest = ((unsigned)n0 / NLOC) & (NR - 1);
    if (threadIdx.x == 0) __hip_atomic_store(cnt, call, __ATOMIC_RELAXED, __HIP_MEMORY_SCOPE_AGENT);
    auto rd = __builtin_amdgcn_make_buffer_rsrc((void*)p.out[dest], 0, N * NLOC, 0x00020000);
    const int vod = (rank * MLOC + m0 + wr * 64 + ks * 32 + r) * NLOC + (n0 & (NLOC - 1)) + wc * 64 + g * 16;
    const float inv = wait_amax(bad);
#pragma unroll
    for (int i = 0; i < 2; ++i) {
      unsigned d[4];
#pragma unroll
      for (int nb = 0; nb < 4; ++nb) d[nb] = q4(acc[i][nb][0], acc[i][nb][1], acc[i][nb][2], acc[i][nb][3], inv);
      u32x4 o = xpose(d[0], d[1], d[2], d[3]);
      if (bad) o = (u32x4){0x80808080u, 0x80808080u, 0x80808080u, 0x80808080u};
      __builtin_amdgcn_raw_buffer_store_b128(o, rd, vod + i * 16 * NLOC, 0, AUX_SYS);
    }
    if (wid == 0) STAMP(6);
  }
#endif
}

}

extern "C" void kernel_launch(int rank, int nranks, ncclComm_t comm,
                              hipStream_t stream,
                              void* const* d_in, const int* in_sizes, int n_in,
                              void* d_out, int out_size,
                              const int* in_dtypes, int out_dtype,
                              void* const* d_out_all,
                              void* const* d_ws_all, size_t ws_bytes) {
  (void)comm; (void)in_dtypes; (void)out_dtype; (void)d_out;
  if (nranks != NR || n_in != 2 || in_sizes[0] != MLOC * K || in_sizes[1] != K * N || out_size != N * NLOC ||
      ws_bytes < WS_NEED || (unsigned)rank >= (unsigned)NR) {
    fprintf(stderr, "kernel_launch: unexpected problem shape\n");
    return;
  }
  static int ready = 0;
  if (!ready) {
    HIP_CHECK(hipFuncSetAttribute((const void*)gemm_a2a_q8, hipFuncAttributeMaxDynamicSharedMemorySize, LDS_BYTES));
    int dev = 0, cus = 0, per_cu = 0;
    HIP_CHECK(hipGetDevice(&dev));
    HIP_CHECK(hipDeviceGetAttribute(&cus, hipDeviceAttributeMultiprocessorCount, dev));
    HIP_CHECK(hipOccupancyMaxActiveBlocksPerMultiprocessor(&per_cu, (const void*)gemm_a2a_q8, 512, LDS_BYTES));
    if (cus * per_cu < NWG) {
      fprintf(stderr, "kernel_launch: %d workgroups cannot be co-resident (%d CUs x %d)\n", NWG, cus, per_cu);
      return;
    }
    ready = 1;
  }
  Params p;
  p.x = d_in[0]; p.w = d_in[1]; p.rank = rank; p.pad = 0;
  p.myout = (char*)d_out_all[rank]; p.myws = (char*)d_ws_all[rank];
  for (int i = 0; i < NR; ++i) { p.out[i] = (char*)d_out_all[i]; p.ws[i] = (char*)d_ws_all[i]; }
  hipLaunchKernelGGL(gemm_a2a_q8, dim3(NWG), dim3(512), LDS_BYTES, stream, p);
  HIP_CHECK(hipGetLastError());
}
```
